```python
import numpy as np

kernel_name = "gpu_a2a_gemm_m4096_k4096_n16384_bf16_silu"


ABSMAX_RTOL = 0.008

COMPUTE_DTYPE = "bfloat16"
EPILOGUE = "silu"
IN_QUANT = None
OUT_QUANT = None
IN_DTYPES = {"x": "bfloat16", "w_mat": "bfloat16"}
WORLD_SIZE = 8

def _to_bf16(x):
    u = np.ascontiguousarray(x, dtype=np.float32).view(np.uint32)
    return ((u + 0x7FFF + ((u >> 16) & 1)) & 0xFFFF0000).view(np.float32)

def setup_inputs():

    x = _to_bf16(np.random.randn(4096, 4096).astype(np.float32))
    w_mat = _to_bf16((np.random.randn(4096, 16384) / np.sqrt(4096)).astype(np.float32))
    return {"x": x, "w_mat": w_mat}

def _epilogue(y):
    return (y / (1.0 + np.exp(-np.clip(y, -60.0, 60.0)))).astype(np.float32)

def reference(x, w_mat):
    y = (x @ w_mat).astype(np.float32)
    return _epilogue(y)
```

```cpp
#include <hip/hip_runtime.h>
#include <rccl/rccl.h>
#include <cstdio>

#define HIP_CHECK(expr) do { hipError_t _e = (expr); if (_e != hipSuccess) { \
    fprintf(stderr, "HIP error %s at %s:%d\n", hipGetErrorString(_e), __FILE__, __LINE__); \
    return; } } while (0)

#define LAS __attribute__((address_space(3)))
typedef short bf16x8 __attribute__((ext_vector_type(8)));
typedef short s16x4 __attribute__((ext_vector_type(4)));
typedef float f32x4 __attribute__((ext_vector_type(4)));
typedef unsigned u32x4 __attribute__((ext_vector_type(4)));

constexpr int NR = 8;
constexpr int NCOL = 16384;
constexpr int KCH = 512;
constexpr int HTB = 128 * 64 * 2;
constexpr int LDS_BYTES = 8 * HTB;
constexpr int NWG = 256;
constexpr int TASYM = 16;
constexpr int NT = 32;

constexpr size_t WS_TMO   = 0;
constexpr size_t WS_CNT   = 4096;
constexpr size_t WS_FLAGA = WS_CNT + NWG * 128;
constexpr size_t WS_FLAGX = WS_FLAGA + 8192;
constexpr size_t WS_RECV  = 1u << 20;
constexpr size_t BLK_BYTES = (size_t)KCH * KCH * 2;
constexpr size_t WS_XCH   = WS_RECV + 2 * NR * BLK_BYTES;
constexpr size_t WS_END   = WS_XCH + (size_t)NWG * 8 * 16 * 1024;
constexpr unsigned long long DEADLINE_TICKS = 40000000ull;

__device__ __forceinline__ int lds_byte(int r, int c) { const int st = (r >> 4) * 2 + (c >> 5), rr = r & 15, cc = c & 31, ob = rr * 64 + cc * 2; return st * 1024 + (ob ^ (((ob >> 9) & 1) << 5)); }
__device__ __forceinline__ void stage_rc(int b, int& R, int& C) { const int st = b / 1024, sb = b % 1024, swz = sb ^ (((sb >> 9) & 1) << 5); R = (st >> 1) * 16 + swz / 64; C = (st & 1) * 32 + (swz % 64) / 2; }
__device__ __forceinline__ unsigned cvt_pk_bf16(float lo, float hi) { unsigned r; asm volatile("v_cvt_pk_bf16_f32 %0, %1, %2" : "=v"(r) : "v"(lo), "v"(hi)); return r; }
__device__ __forceinline__ float silu(float y) {
    const float c = __builtin_amdgcn_fmed3f(y, -60.0f, 60.0f);
    const float e = __builtin_amdgcn_exp2f(c * -1.44269504088896f);
    return y * __builtin_amdgcn_rcpf(1.0f + e);
}
__device__ __forceinline__ unsigned vm_outstanding() {
    const unsigned v = (unsigned)__builtin_amdgcn_s_getreg((31 << 11) | 7);
    return (v & 0xFu) | ((v >> 18) & 0x30u);
}

#define STAMP(k) do {} while (0)
#define STAMPC(k) do {} while (0)
#define WAIT_V(n) asm volatile("s_waitcnt vmcnt(" #n ")" ::: "memory")
#define WAIT_L(n) asm volatile("s_waitcnt lgkmcnt(" #n ")" ::: "memory")
#define BAR __builtin_amdgcn_s_barrier()
#define SCHED __builtin_amdgcn_sched_barrier(0)

__device__ __forceinline__ bool wait_flags(const unsigned* f, unsigned call, unsigned* tmo) {
    const unsigned long long t0 = __builtin_amdgcn_s_memrealtime();
    for (;;) {
        const unsigned v = __hip_atomic_load(f, __ATOMIC_RELAXED, __HIP_MEMORY_SCOPE_SYSTEM);
        const bool ok = (int)(v - call) >= 0;
        if (__builtin_amdgcn_ballot_w64(!ok) == 0ull) return true;
        if ((unsigned)__builtin_amdgcn_readfirstlane(__hip_atomic_load(tmo, __ATOMIC_RELAXED, __HIP_MEMORY_SCOPE_AGENT)) == call ||
            __builtin_amdgcn_s_memrealtime() - t0 > DEADLINE_TICKS) {
            __hip_atomic_store(tmo, call, __ATOMIC_RELAXED, __HIP_MEMORY_SCOPE_AGENT);
            return false;
        }
        __builtin_amdgcn_s_sleep(1);
    }
}

__global__ __launch_bounds__(512, 2)
void a2a_gemm_silu(const char* __restrict__ x, const char* __restrict__ w, char* __restrict__ out,
                   void* const* __restrict__ ws_all, int rank) {
    extern __shared__ __attribute__((aligned(16))) unsigned char shm[];
    LAS unsigned char* lds = (LAS unsigned char*)shm;
    const int tid = threadIdx.x, wid = __builtin_amdgcn_readfirstlane(tid >> 6), lane = tid & 63;
    const int wr = wid >> 2, wc = wid & 3, fr = lane & 15, fq = lane >> 4;
    const int b = blockIdx.x, idx = b >> 3;
    const int ntile = (b & 7) * 8 + (idx >> 2), m = idx & 1, h = (idx >> 1) & 1;
    char* ws = (char*)ws_all[rank];
    unsigned* tmo = (unsigned*)(ws + WS_TMO);
    STAMP(0);

    unsigned* cnt = (unsigned*)(ws + WS_CNT + (size_t)b * 128);
    unsigned cntv;
    asm volatile("global_load_dword %0, %1, off sc1" : "=v"(cntv) : "v"(cnt) : "memory");

    unsigned voffA[2], voffB[2];
#pragma unroll
    for (int i = 0; i < 2; ++i) {
        int R, C; stage_rc(tid * 16 + i * 8192, R, C);
        voffA[i] = (unsigned)(R * KCH + C) * 2u;
        const int r = wid * 4 + (lane >> 4) + i * 32, c = (lane & 15) ^ (((r & 3) << 2) | ((r >> 3) & 1));
        voffB[i] = (unsigned)r * (unsigned)(NCOL * 2) + (unsigned)c * 16u;
    }
    const unsigned ldsw = (unsigned)wid * 1024u;
    const int aoff = lds_byte(wr * 64 + fr, fq * 8);
    const int q4 = (lane & 15) >> 2, p4 = lane & 3;
    const int boff0 = 256 * (8 * fq + q4) + 16 * ((4 * wc + p4) ^ (q4 << 2) ^ (fq & 1)) + 8 * (p4 & 1);
    const unsigned bad0 = (unsigned)(size_t)lds + 65536u + (unsigned)boff0, bad1 = bad0 ^ 8u;

    const int flip = h ^ 1;
    const long aoffs0 = (long)(m * 256 + flip * 128) * (KCH * 2) + h * 512;
    const long hstepA = (flip ? -1L : 1L) * 128 * (KCH * 2);
    const char* xloc = x + (size_t)rank * BLK_BYTES + aoffs0;
    const char* wbase = w + (size_t)(h * 256) * (NCOL * 2) + (size_t)ntile * 512;
    constexpr long hstepB = 256;

    const int coff = ((h * 2 + m) * 7) >> 2;
#define TILE_PTRS(s_, A_, B_) do { const int _s = (s_) & 31; const int _u = _s < 4 ? 0 : _s - 4; const int _j0 = (_u * 37) >> 8; \
        const int _d0 = _u - _j0 * 7 + coff; const int _d = _d0 >= 7 ? _d0 - 7 : _d0; \
        const int _j = _s < 4 ? _s : _j0; const int _p = _s < 4 ? rank : ((rank + 1 + _d) & 7); \
        A_ = _s < 4 ? xloc + _j * 128 : xrem + (size_t)_j * BLK_BYTES + _p * 128; \
        B_ = wbase + (size_t)(_p * KCH + _j * 64) * (NCOL * 2); } while (0)

#define SA(bb, hh) (((bb) * 2 + (hh)) * HTB)
#define SB(bb, hh) ((4 + (bb) * 2 + (hh)) * HTB)
#define GLDS(gp, loff) __builtin_amdgcn_global_load_lds((const unsigned*)(gp), (LAS unsigned*)(lds + (loff)), 16, 0, 0)
#define STAGE(ASYM, bufoff, gbase, voff, d7) do { \
        if (!(ASYM) || wid != 7) { GLDS((gbase) + (voff)[0], (bufoff) + ldsw); GLDS((gbase) + (voff)[1], (bufoff) + ldsw + 8192); } \
        if ((ASYM) && wid == 6) { GLDS((gbase) + (d7) + (voff)[0], (bufoff) + 7 * 1024); GLDS((gbase) + (d7) + (voff)[1], (bufoff) + 7 * 1024 + 8192); } } while (0)
#define STA(ASYM, bb, hh, ap) STAGE(ASYM, SA(bb, hh), (ap) + (hh) * hstepA, voffA, 64)
#define STB(ASYM, bb, hh, bp) STAGE(ASYM, SB(bb, hh), (bp) + (hh) * hstepB, voffB, 4 * NCOL * 2)
#define WAITV(ASYM, n1, n2) do { if (!(ASYM)) WAIT_V(n1); else if (wid == 6) WAIT_V(n2); else if (wid != 7) WAIT_V(n1); } while (0)
#define LDA(dst, bb, hh) do { _Pragma("unroll") for (int mm = 0; mm < 4; ++mm) _Pragma("unroll") for (int k = 0; k < 2; ++k) \
        dst[mm][k] = *(const LAS bf16x8*)(lds + SA(bb, hh) + aoff + mm * 2048 + k * 1024); } while (0)
#define TRD(dst, base, off) asm volatile("ds_read_b64_tr_b16 %0, %1 offset:%2" : "=v"(dst) : "v"(base), "n"(off) : "memory")
#define LDB(dst, bb, hh) do { _Pragma("unroll") for (int n = 0; n < 2; ++n) _Pragma("unroll") for (int k = 0; k < 2; ++k) { \
        s16x4 _lo, _hi; TRD(_lo, (n ? bad1 : bad0), SB(bb, hh) - 65536 + k * 8192); TRD(_hi, (n ? bad1 : bad0), SB(bb, hh) - 65536 + k * 8192 + 1024); \
        dst[n][k] = __builtin_shufflevector(_lo, _hi, 0, 1, 2, 3, 4, 5, 6, 7); } } while (0)
#define WAIT_LB(Bt_) asm volatile("s_waitcnt lgkmcnt(0)" : "+v"(Bt_[0][0]), "+v"(Bt_[0][1]), "+v"(Bt_[1][0]), "+v"(Bt_[1][1]) :: "memory")
#define MMA(ai, bj, At_, Bt_) do { __builtin_amdgcn_s_setprio(1); _Pragma("unroll") for (int mm = 0; mm < 4; ++mm) _Pragma("unroll") for (int n = 0; n < 2; ++n) _Pragma("unroll") for (int k = 0; k < 2; ++k) \
        acc[ai][bj][mm][n] = __builtin_amdgcn_mfma_f32_16x16x32_bf16(Bt_[n][k], At_[mm][k], acc[ai][bj][mm][n], 0, 0, 0); __builtin_amdgcn_s_setprio(0); } while (0)

#define EARLYB 0
#define MMAB(ai, bj, At_, Bt_) do { __builtin_amdgcn_s_setprio(1); _Pragma("unroll") for (int mm = 0; mm < 4; ++mm) { \
        if (mm == 4 - EARLYB) { SCHED; BAR; SCHED; } \
        _Pragma("unroll") for (int n = 0; n < 2; ++n) _Pragma("unroll") for (int k = 0; k < 2; ++k) \
        acc[ai][bj][mm][n] = __builtin_amdgcn_mfma_f32_16x16x32_bf16(Bt_[n][k], At_[mm][k], acc[ai][bj][mm][n], 0, 0, 0); } \
        __builtin_amdgcn_s_setprio(0); if (EARLYB == 0) BAR; SCHED; } while (0)

    const char *a0 = xloc, *b0 = wbase + (size_t)(rank * KCH) * (NCOL * 2), *a1 = a0 + 128, *b1 = b0 + (size_t)64 * (NCOL * 2);
    const bool pusher = wid == 7 && b < 224;
    const int ph = (b >> 4) & 1, rg = b & 15;
    const unsigned dst = (unsigned)(rank + 1 + (b >> 5)) & 7u;
    const unsigned loff = (unsigned)(rg * 32 + (lane >> 3)) * (KCH * 2) + (lane & 7) * 16;
    const char* psrc = x + (size_t)dst * BLK_BYTES + ph * 512;
    u32x4 pv[4];
#define PUSH_LOAD(j_) do { _Pragma("unroll") for (int i = 0; i < 4; ++i) \
        asm volatile("global_load_dwordx4 %0, %1, %2 nt" : "=v"(pv[i]) : "v"(loff), "s"(psrc + i * 8 * (KCH * 2) + (j_) * 128) : "memory"); } while (0)
#define PUSH_STORE(j_) do { _Pragma("unroll") for (int i = 0; i < 4; ++i) \
        asm volatile("global_store_dwordx4 %0, %1, %2 sc0 sc1\n\ts_nop 2" :: "v"(loff), "v"(pv[i]), "s"(pdst + i * 8 * (KCH * 2) + (size_t)(j_) * BLK_BYTES) : "memory"); } while (0)
    if (pusher) PUSH_LOAD(0);
    STB(1, 0, 0, b0); STA(1, 0, 0, a0); STB(1, 0, 1, b0); STA(1, 0, 1, a0);
    char* pws = (char*)ws_all[dst];
    if (wid == 7) WAIT_V(0); else if (wid == 6) WAIT_V(16); else WAIT_V(8);
    asm volatile("" : "+v"(cntv));
    const unsigned call = (unsigned)__builtin_amdgcn_readfirstlane(cntv) + 1u;
    const unsigned par = call & 1u;
    const char* xrem = ws + WS_RECV + (size_t)(par * 8 + h * 4) * BLK_BYTES + (size_t)(m * 256 + flip * 128) * (KCH * 2);
    char* pdst = pws + WS_RECV + (size_t)(par * 8 + ph * 4) * BLK_BYTES + rank * 128;
    const int pcl = ph * 2 + (rg >> 3);
    const int uc0 = 6 - (b >> 5) - ((pcl * 7) >> 2), uc = uc0 < 0 ? uc0 + 7 : uc0;
    unsigned* pflag = (unsigned*)(pws + WS_FLAGA) + pcl * 512 + (rg & 7);
    int ps = -1;
    int sn = pusher ? 4 + uc : 99;
    int dl = (uc + 1) >> 1;
#define PUSH_ADVANCE() do { \
        if (ps >= 0) { if (lane == 0) asm volatile("global_store_dword %0, %1, off sc0 sc1" :: "v"(pflag + sn * 8), "v"(call) : "memory"); sn += 7; } \
        if (ps < 3) PUSH_STORE(ps + 1); \
        if (ps < 2) PUSH_LOAD(ps + 2); \
        ++ps; } while (0)
    if (sn < 32 && dl == 0) PUSH_ADVANCE();
#define PUSH_POLL() do { if (sn < 32) { --dl; if (dl <= 0 && vm_outstanding() == 0u) PUSH_ADVANCE(); } } while (0)
#define PUSH_FLUSH(lim) do { while (sn <= (lim)) { WAIT_V(0); PUSH_ADVANCE(); } } while (0)

    f32x4 acc[2][2][4][2];
#pragma unroll
    for (int a = 0; a < 2; ++a)
#pragma unroll
        for (int c = 0; c < 2; ++c)
#pragma unroll
            for (int mm = 0; mm < 4; ++mm)
#pragma unroll
                for (int n = 0; n < 2; ++n) acc[a][c][mm][n] = (f32x4){0.f, 0.f, 0.f, 0.f};
    bf16x8 At[4][2], B0[2][2], B1[2][2];
    bool good = true;
    const char* fbase = ws + WS_FLAGA + (h * 2 + m) * 2048;
    unsigned fo = (unsigned)(lane & 7) * 4u, fva = 0u, fvb = 0u;
#define FLAG_ISSUE(fv, ds) do { if (wr == 0) asm volatile("global_load_dword %0, %1, %2 offset:%3 sc0 sc1" : "=v"(fv) : "v"(fo), "s"(fbase), "n"((ds) * 32) : "memory"); } while (0)
#define FLAG_CHECK(fv, ds) do { if (wr == 0 && t >= 2) { asm volatile("" : "+v"(fv)); \
        if (__builtin_amdgcn_ballot_w64((int)(fv - call) < 0) != 0ull) good = wait_flags((const unsigned*)(fbase + fo + (ds) * 32), call, tmo) && good; } } while (0)

    const char *a2, *b2, *a3, *b3; TILE_PTRS(2, a2, b2); TILE_PTRS(3, a3, b3);
    if (wr == 1) BAR;
    WAITV(1, 4, 8); BAR;
    STB(1, 1, 0, b1); STA(1, 1, 0, a1); STB(1, 1, 1, b1);
    WAITV(1, 6, 12); BAR;
    STAMP(1);

    LDB(B0, 0, 0);
#define ITER(ASYM) do { \
        FLAG_ISSUE(fvb, 3); LDA(At, 0, 0); STA(ASYM, 1, 1, a1); \
        WAIT_L(8); BAR; WAIT_LB(B0); MMAB(0, 0, At, B0); \
        LDB(B1, 0, 1); STB(ASYM, 0, 0, b2); if (ASYM) { PUSH_POLL(); PUSH_FLUSH(t + 2); } \
        BAR; WAIT_LB(B1); MMAB(0, 1, At, B1); \
        LDA(At, 0, 1); FLAG_CHECK(fva, 2); STA(ASYM, 0, 0, a2); \
        WAITV(ASYM, 10, 20); BAR; WAIT_L(0); MMAB(1, 0, At, B0); \
        LDB(B0, 1, 0); STB(ASYM, 0, 1, b2); if (ASYM) PUSH_POLL(); \
        WAITV(ASYM, 6, 12); BAR; MMAB(1, 1, At, B1); \
        FLAG_ISSUE(fva, 4); LDA(At, 1, 0); STA(ASYM, 0, 1, a2); \
        WAIT_L(8); BAR; WAIT_LB(B0); MMAB(0, 0, At, B0); \
        LDB(B1, 1, 1); STB(ASYM, 1, 0, b3); if (ASYM) { PUSH_POLL(); PUSH_FLUSH(t + 3); } \
        BAR; WAIT_LB(B1); MMAB(0, 1, At, B1); \
        LDA(At, 1, 1); FLAG_CHECK(fvb, 3); STA(ASYM, 1, 0, a3); \
        WAITV(ASYM, 10, 20); BAR; WAIT_L(0); MMAB(1, 0, At, B0); \
        LDB(B0, 0, 0); STB(ASYM, 1, 1, b3); if (ASYM) { PUSH_POLL(); if (t == TASYM - 2) PUSH_FLUSH(31); } \
        a1 = a3; TILE_PTRS(t + 4, a2, b2); TILE_PTRS(t + 5, a3, b3); fo += 64u; SCHED; \
        WAITV(ASYM, 6, 12); BAR; MMAB(1, 1, At, B1); \
        } while (0)

#pragma unroll 1
    for (int t = 0; t < TASYM; t += 2) {
        if (t == 2) STAMP(2);
        if (t == 4) STAMP(3);
        if (t == 6) STAMP(4);
        if (t == 8) STAMP(5);
        ITER(1);
    }
#pragma unroll 1
    for (int t = TASYM; t < NT - 2; t += 2) {
        if (t == 16) STAMP(6);
        if (t == 20) { STAMP(7); STAMPC(12); }
        ITER(0);
    }
    STAMP(8); STAMPC(13);

    int tz = threadIdx.x; asm volatile("" : "+v"(tz));
    const int lane2 = tz & 63, fr2 = lane2 & 15, fq2 = lane2 >> 4;
    {
        char* xs = ws + WS_XCH + ((size_t)b * 8 + wid) * 8192 + lane2 * 16;
#define XSEND(c) do { _Pragma("unroll") for (int mm = 0; mm < 4; ++mm) { \
            const f32x4 u0 = acc[0][c][mm][0], u1 = acc[0][c][mm][1]; \
            const u32x4 o = {cvt_pk_bf16(u0[0], u0[1]), cvt_pk_bf16(u0[2], u0[3]), cvt_pk_bf16(u1[0], u1[1]), cvt_pk_bf16(u1[2], u1[3])}; \
            asm volatile("global_store_dwordx4 %0, %1, off offset:%2 sc1\n\ts_nop 2" :: "v"(xs + (c) * 4096), "v"(o), "n"(mm * 1024) : "memory"); } } while (0)
        LDA(At, 0, 0); STA(0, 1, 1, a1);
        BAR; WAIT_LB(B0); MMAB(0, 0, At, B0);
        LDB(B1, 0, 1);
        BAR; WAIT_LB(B1); MMAB(0, 1, At, B1);
        LDA(At, 0, 1);
        WAIT_V(6); BAR; WAIT_L(0); MMAB(1, 0, At, B0);
        LDB(B0, 1, 0);
        WAIT_V(0); BAR; MMAB(1, 1, At, B1);
        LDA(At, 1, 0);
        BAR; WAIT_LB(B0); MMAB(0, 0, At, B0);
        LDB(B1, 1, 1); XSEND(0); SCHED;
        BAR; WAIT_LB(B1); MMAB(0, 1, At, B1);
        LDA(At, 1, 1); XSEND(1); SCHED;
        BAR; WAIT_L(0); MMA(1, 0, At, B0); if (wr == 0) BAR;
        SCHED; MMA(1, 1, At, B1); SCHED;
        WAIT_V(0);
        if (lane2 == 0) __hip_atomic_store((unsigned*)(ws + WS_FLAGX + ((size_t)b * 8 + wid) * 128), call, __ATOMIC_RELAXED, __HIP_MEMORY_SCOPE_AGENT);
    }
    if (tz == 0) __hip_atomic_store(cnt, call, __ATOMIC_RELAXED, __HIP_MEMORY_SCOPE_AGENT);

    STAMP(9);
    {
        const int pb = b ^ 16;
        const unsigned* xf = (const unsigned*)(ws + WS_FLAGX + ((size_t)pb * 8 + wid) * 128);
        const unsigned long long t0 = __builtin_amdgcn_s_memrealtime();
        while ((int)((unsigned)__builtin_amdgcn_readfirstlane(__hip_atomic_load(xf, __ATOMIC_RELAXED, __HIP_MEMORY_SCOPE_AGENT)) - call) < 0) {
            if ((unsigned)__builtin_amdgcn_readfirstlane(__hip_atomic_load(tmo, __ATOMIC_RELAXED, __HIP_MEMORY_SCOPE_AGENT)) == call ||
                __builtin_amdgcn_s_memrealtime() - t0 > DEADLINE_TICKS) {
                __hip_atomic_store(tmo, call, __ATOMIC_RELAXED, __HIP_MEMORY_SCOPE_AGENT);
                break;
            }
            __builtin_amdgcn_s_sleep(1);
        }
        STAMP(10);
        const char* xr = ws + WS_XCH + ((size_t)pb * 8 + wid) * 8192 + lane2 * 16;
        u32x4 rv[2][4];
#pragma unroll
        for (int mm = 0; mm < 4; ++mm)
#pragma unroll
            for (int c = 0; c < 2; ++c)
                asm volatile("global_load_dwordx4 %0, %1, off sc1" : "=v"(rv[c][mm]) : "v"(xr + (c * 4 + mm) * 1024) : "memory");
        const bool bad = !good || (unsigned)__builtin_amdgcn_readfirstlane(__hip_atomic_load(tmo, __ATOMIC_RELAXED, __HIP_MEMORY_SCOPE_AGENT)) == call;
        WAIT_V(0);
#pragma unroll
        for (int c = 0; c < 2; ++c)
#pragma unroll
            for (int mm = 0; mm < 4; ++mm) asm volatile("" : "+v"(rv[c][mm]));
        char* op = out + (size_t)(m * 256 + h * 128 + wr * 64 + fr2) * (NCOL * 2) + (size_t)(ntile * 256 + wc * 32 + 8 * fq2) * 2;
        const bool odd = fq2 & 1;
#pragma unroll
        for (int mm = 0; mm < 4; ++mm)
#pragma unroll
            for (int c = 0; c < 2; ++c) {
                const u32x4 r = rv[c][mm];
                const f32x4 v0 = acc[1][c][mm][0] + (f32x4){__uint_as_float(r[0] << 16), __uint_as_float(r[0] & 0xFFFF0000u), __uint_as_float(r[1] << 16), __uint_as_float(r[1] & 0xFFFF0000u)};
                const f32x4 v1 = acc[1][c][mm][1] + (f32x4){__uint_as_float(r[2] << 16), __uint_as_float(r[2] & 0xFFFF0000u), __uint_as_float(r[3] << 16), __uint_as_float(r[3] & 0xFFFF0000u)};
                const unsigned l0 = cvt_pk_bf16(silu(v0[0]), silu(v0[1])), l1 = cvt_pk_bf16(silu(v0[2]), silu(v0[3]));
                const unsigned h0 = cvt_pk_bf16(silu(v1[0]), silu(v1[1])), h1 = cvt_pk_bf16(silu(v1[2]), silu(v1[3]));
                u32x4 o = odd ? (u32x4){h0, h1, l0, l1} : (u32x4){l0, l1, h0, h1};
                if (bad) o = (u32x4){0x7FC07FC0u, 0x7FC07FC0u, 0x7FC07FC0u, 0x7FC07FC0u};
                *(u32x4*)(op + (size_t)mm * 16 * (NCOL * 2) + c * 256) = o;
            }
        STAMP(11);
    }
}

extern "C" void kernel_launch(int rank, int nranks, ncclComm_t comm,
                              hipStream_t stream,
                              void* const* d_in, const int* in_sizes, int n_in,
                              void* d_out, int out_size,
                              const int* in_dtypes, int out_dtype,
                              void* const* d_out_all,
                              void* const* d_ws_all, size_t ws_bytes) {
    if (nranks != NR || (unsigned)rank >= (unsigned)NR || ws_bytes < WS_END) { fprintf(stderr, "unsupported configuration\n"); return; }
    static bool init = false;
    if (!init) {
        HIP_CHECK(hipFuncSetAttribute((const void*)a2a_gemm_silu, hipFuncAttributeMaxDynamicSharedMemorySize, LDS_BYTES));
        init = true;
    }
    hipLaunchKernelGGL(a2a_gemm_silu, dim3(NWG), dim3(512), LDS_BYTES, stream,
                       (const char*)d_in[0], (const char*)d_in[1], (char*)d_out, d_ws_all, rank);
    HIP_CHECK(hipGetLastError());
}
```

```cpp
#include <hip/hip_runtime.h>
#include <rccl/rccl.h>
#include <cstdio>

#define HIP_CHECK(expr) do { hipError_t _e = (expr); if (_e != hipSuccess) { \
    fprintf(stderr, "HIP error %s at %s:%d\n", hipGetErrorString(_e), __FILE__, __LINE__); \
    return; } } while (0)

#define LAS __attribute__((address_space(3)))
typedef short bf16x8 __attribute__((ext_vector_type(8)));
typedef short s16x4 __attribute__((ext_vector_type(4)));
typedef float f32x4 __attribute__((ext_vector_type(4)));
typedef unsigned u32x4 __attribute__((ext_vector_type(4)));

constexpr int NR = 8;
constexpr int NCOL = 16384;
constexpr int KCH = 512;
constexpr int HTB = 128 * 64 * 2;
constexpr int LDS_BYTES = 8 * HTB;
constexpr int NWG = 256;
constexpr int TPUSH = 10;
constexpr int NT = 32;

constexpr size_t WS_TMO   = 0;
constexpr size_t WS_CNT   = 4096;
constexpr size_t WS_FLAGA = WS_CNT + NWG * 128;
constexpr size_t WS_FLAGX = WS_FLAGA + 32768;
constexpr size_t WS_RECV  = 1u << 20;
constexpr size_t BLK_BYTES = (size_t)KCH * KCH * 2;
constexpr size_t WS_XCH   = WS_RECV + 2 * NR * BLK_BYTES;
constexpr size_t WS_END   = WS_XCH + (size_t)NWG * 8 * 16 * 1024;
constexpr unsigned long long DEADLINE_TICKS = 40000000ull;

__device__ __forceinline__ int lds_byte(int r, int c) { const int st = (r >> 4) * 2 + (c >> 5), rr = r & 15, cc = c & 31, ob = rr * 64 + cc * 2; return st * 1024 + (ob ^ (((ob >> 9) & 1) << 5)); }
__device__ __forceinline__ void stage_rc(int b, int& R, int& C) { const int st = b / 1024, sb = b % 1024, swz = sb ^ (((sb >> 9) & 1) << 5); R = (st >> 1) * 16 + swz / 64; C = (st & 1) * 32 + (swz % 64) / 2; }
__device__ __forceinline__ unsigned cvt_pk_bf16(float lo, float hi) { unsigned r; asm volatile("v_cvt_pk_bf16_f32 %0, %1, %2" : "=v"(r) : "v"(lo), "v"(hi)); return r; }
__device__ __forceinline__ float silu(float y) {
    const float c = __builtin_amdgcn_fmed3f(y, -60.0f, 60.0f);
    const float e = __builtin_amdgcn_exp2f(c * -1.44269504088896f);
    return y * __builtin_amdgcn_rcpf(1.0f + e);
}

#define STAMP(k) do {} while (0)
#define STAMPC(k) do {} while (0)
#define WAIT_V(n) asm volatile("s_waitcnt vmcnt(" #n ")" ::: "memory")
#define WAIT_L(n) asm volatile("s_waitcnt lgkmcnt(" #n ")" ::: "memory")
#define BAR __builtin_amdgcn_s_barrier()
#define SCHED __builtin_amdgcn_sched_barrier(0)

__device__ __forceinline__ bool wait_flags(const unsigned* f, unsigned call, unsigned* tmo) {
    const unsigned long long t0 = __builtin_amdgcn_s_memrealtime();
    for (;;) {
        const unsigned v = __hip_atomic_load(f, __ATOMIC_RELAXED, __HIP_MEMORY_SCOPE_SYSTEM);
        const bool ok = (int)(v - call) >= 0;
        if (__builtin_amdgcn_ballot_w64(!ok) == 0ull) return true;
        if ((unsigned)__builtin_amdgcn_readfirstlane(__hip_atomic_load(tmo, __ATOMIC_RELAXED, __HIP_MEMORY_SCOPE_AGENT)) == call ||
            __builtin_amdgcn_s_memrealtime() - t0 > DEADLINE_TICKS) {
            __hip_atomic_store(tmo, call, __ATOMIC_RELAXED, __HIP_MEMORY_SCOPE_AGENT);
            return false;
        }
        __builtin_amdgcn_s_sleep(1);
    }
}

__global__ __launch_bounds__(512, 2)
void a2a_gemm_silu(const char* __restrict__ x, const char* __restrict__ w, char* __restrict__ out,
                   void* const* __restrict__ ws_all, int rank) {
    extern __shared__ __attribute__((aligned(16))) unsigned char shm[];
    LAS unsigned char* lds = (LAS unsigned char*)shm;
    const int tid = threadIdx.x, wid = __builtin_amdgcn_readfirstlane(tid >> 6), lane = tid & 63;
    const int wr = wid >> 2, wc = wid & 3, fr = lane & 15, fq = lane >> 4;
    const int b = blockIdx.x, idx = b >> 3;
    const int ntile = (b & 7) * 8 + (idx >> 2), m = idx & 1, h = (idx >> 1) & 1;
    char* ws = (char*)ws_all[rank];
    unsigned* tmo = (unsigned*)(ws + WS_TMO);
    STAMP(0);

    unsigned* cnt = (unsigned*)(ws + WS_CNT + (size_t)b * 128);
    unsigned cntv;
    asm volatile("global_load_dword %0, %1, off sc1" : "=v"(cntv) : "v"(cnt) : "memory");

    unsigned voffA[2], voffB[2];
#pragma unroll
    for (int i = 0; i < 2; ++i) {
        int R, C; stage_rc(tid * 16 + i * 8192, R, C);
        voffA[i] = (unsigned)(R * KCH + C) * 2u;
        const int r = wid * 4 + (lane >> 4) + i * 32, c = (lane & 15) ^ (((r & 3) << 2) | ((r >> 3) & 1));
        voffB[i] = (unsigned)r * (unsigned)(NCOL * 2) + (unsigned)c * 16u;
    }
    const unsigned ldsw = (unsigned)wid * 1024u;
    const int aoff = lds_byte(wr * 64 + fr, fq * 8);
    const int q4 = (lane & 15) >> 2, p4 = lane & 3;
    const int boff0 = 256 * (8 * fq + q4) + 16 * ((4 * wc + p4) ^ (q4 << 2) ^ (fq & 1)) + 8 * (p4 & 1);
    const unsigned bad0 = (unsigned)(size_t)lds + 65536u + (unsigned)boff0, bad1 = bad0 ^ 8u;

    const int flip = h ^ 1;
    const long aoffs0 = (long)(m * 256 + flip * 128) * (KCH * 2) + h * 512;
    const long hstepA = (flip ? -1L : 1L) * 128 * (KCH * 2);
    const char* xloc = x + (size_t)rank * BLK_BYTES + aoffs0;
    const char* wbase = w + (size_t)(h * 256) * (NCOL * 2) + (size_t)ntile * 512;
    constexpr long hstepB = 256;

    const int coff = ((h * 2 + m) * 7) >> 2;
#define TILE_PTRS(s_, A_, B_) do { const int _s = (s_) & 31; const int _u = _s < 4 ? 0 : _s - 4; const int _j0 = (_u * 37) >> 8; \
        const int _d0 = _u - _j0 * 7 + coff; const int _d = _d0 >= 7 ? _d0 - 7 : _d0; \
        const int _j = _s < 4 ? _s : _j0; const int _p = _s < 4 ? rank : ((rank + 1 + _d) & 7); \
        A_ = _s < 4 ? xloc + _j * 128 : xrem + (size_t)_j * BLK_BYTES + _p * 128; \
        B_ = wbase + (size_t)(_p * KCH + _j * 64) * (NCOL * 2); } while (0)

#define SA(bb, hh) (((bb) * 2 + (hh)) * HTB)
#define SB(bb, hh) ((4 + (bb) * 2 + (hh)) * HTB)
#define GLDS(gp, loff) __builtin_amdgcn_global_load_lds((const unsigned*)(gp), (LAS unsigned*)(lds + (loff)), 16, 0, 0)
#define STAGE(bufoff, gbase, voff) do { GLDS((gbase) + (voff)[0], (bufoff) + ldsw); GLDS((gbase) + (voff)[1], (bufoff) + ldsw + 8192); } while (0)
#define STA(bb, hh, ap) STAGE(SA(bb, hh), (ap) + (hh) * hstepA, voffA)
#define STB(bb, hh, bp) STAGE(SB(bb, hh), (bp) + (hh) * hstepB, voffB)
#define LDA(dst, bb, hh) do { _Pragma("unroll") for (int mm = 0; mm < 4; ++mm) _Pragma("unroll") for (int k = 0; k < 2; ++k) \
        dst[mm][k] = *(const LAS bf16x8*)(lds + SA(bb, hh) + aoff + mm * 2048 + k * 1024); } while (0)
#define TRD(dst, base, off) asm volatile("ds_read_b64_tr_b16 %0, %1 offset:%2" : "=v"(dst) : "v"(base), "n"(off) : "memory")
#define LDB(dst, bb, hh) do { _Pragma("unroll") for (int n = 0; n < 2; ++n) _Pragma("unroll") for (int k = 0; k < 2; ++k) { \
        s16x4 _lo, _hi; TRD(_lo, (n ? bad1 : bad0), SB(bb, hh) - 65536 + k * 8192); TRD(_hi, (n ? bad1 : bad0), SB(bb, hh) - 65536 + k * 8192 + 1024); \
        dst[n][k] = __builtin_shufflevector(_lo, _hi, 0, 1, 2, 3, 4, 5, 6, 7); } } while (0)
#define WAIT_LB(Bt_) asm volatile("s_waitcnt lgkmcnt(0)" : "+v"(Bt_[0][0]), "+v"(Bt_[0][1]), "+v"(Bt_[1][0]), "+v"(Bt_[1][1]) :: "memory")
#define MMA(ai, bj, At_, Bt_) do { __builtin_amdgcn_s_setprio(1); _Pragma("unroll") for (int mm = 0; mm < 4; ++mm) _Pragma("unroll") for (int n = 0; n < 2; ++n) _Pragma("unroll") for (int k = 0; k < 2; ++k) \
        acc[ai][bj][mm][n] = __builtin_amdgcn_mfma_f32_16x16x32_bf16(Bt_[n][k], At_[mm][k], acc[ai][bj][mm][n], 0, 0, 0); __builtin_amdgcn_s_setprio(0); } while (0)

#define EARLYB 0
#define MMAB(ai, bj, At_, Bt_) do { __builtin_amdgcn_s_setprio(1); _Pragma("unroll") for (int mm = 0; mm < 4; ++mm) { \
        if (mm == 4 - EARLYB) { SCHED; BAR; SCHED; } \
        _Pragma("unroll") for (int n = 0; n < 2; ++n) _Pragma("unroll") for (int k = 0; k < 2; ++k) \
        acc[ai][bj][mm][n] = __builtin_amdgcn_mfma_f32_16x16x32_bf16(Bt_[n][k], At_[mm][k], acc[ai][bj][mm][n], 0, 0, 0); } \
        __builtin_amdgcn_s_setprio(0); if (EARLYB == 0) BAR; SCHED; } while (0)

    const char *a0 = xloc, *b0 = wbase + (size_t)(rank * KCH) * (NCOL * 2), *a1 = a0 + 128, *b1 = b0 + (size_t)64 * (NCOL * 2);
    const bool pusher = b < 224;
    const int k1 = (b >> 2) & 7;
    const unsigned dst = (unsigned)(rank + 1 + (b >> 5)) & 7u;
    int pcl = 0, uc = 0;
    {
        int u[4];
#pragma unroll
        for (int c = 0; c < 4; ++c) { const int v = 6 - (b >> 5) - ((c * 7) >> 2); u[c] = v < 0 ? v + 7 : v; }
#pragma unroll
        for (int c = 0; c < 4; ++c) {
            const int o = (u[0] < u[c]) + (u[1] < u[c]) + (u[2] < u[c]) + (u[3] < u[c]);
            if (o == (k1 & 3)) { pcl = c; uc = u[c]; }
        }
    }
    const int pj = k1 >> 2;
    const unsigned loff = (unsigned)((pcl & 1) * 256 + (b & 3) * 64 + wid * 8 + (lane >> 3)) * (KCH * 2) + (lane & 7) * 16;
    const char* psrc = x + (size_t)dst * BLK_BYTES + (pcl >> 1) * 512 + pj * 128;
    u32x4 pv;
    if (pusher) asm volatile("global_load_dwordx4 %0, %1, %2 nt" : "=v"(pv) : "v"(loff), "s"(psrc) : "memory");
    STB(0, 0, b0); STA(0, 0, a0); STB(0, 1, b0); STA(0, 1, a0); STB(1, 0, b1); STA(1, 0, a1); STB(1, 1, b1);
    char* pws = (char*)ws_all[dst];
    WAIT_V(14);
    asm volatile("" : "+v"(cntv));
    const unsigned call = (unsigned)__builtin_amdgcn_readfirstlane(cntv) + 1u;
    const unsigned par = call & 1u;
    const char* xrem = ws + WS_RECV + (size_t)(par * 8 + h * 4) * BLK_BYTES + (size_t)(m * 256 + flip * 128) * (KCH * 2);
    char* pdst = pws + WS_RECV + (size_t)(par * 8 + (pcl >> 1) * 4 + pj) * BLK_BYTES + rank * 128;
    char* pflag = pws + WS_FLAGA + (size_t)(pcl * 64 + 4 + 7 * pj + uc) * 128 + ((b & 3) * 8 + wid) * 4;
    int ps = 0, ev = pusher ? k1 + 1 : 1000;
#define PUSH_SLOT() do { if (--ev == 0) { \
        if (ps & 1) { WAIT_V(0); if (lane == 0) asm volatile("global_store_dword %0, %1, %2 sc0 sc1" :: "v"(0u), "v"(call), "s"(pflag) : "memory"); pflag += 14 * 128; } \
        else { asm volatile("global_store_dwordx4 %0, %1, %2 sc0 sc1\n\ts_nop 2" :: "v"(loff), "v"(pv), "s"(pdst) : "memory"); pdst += 2 * BLK_BYTES; \
               if (ps == 0) asm volatile("global_load_dwordx4 %0, %1, %2 offset:256 nt" : "=v"(pv) : "v"(loff), "s"(psrc) : "memory"); } \
        ++ps; ev = ps < 4 ? 4 : 1000; } } while (0)
    PUSH_SLOT();

    f32x4 acc[2][2][4][2];
#pragma unroll
    for (int a = 0; a < 2; ++a)
#pragma unroll
        for (int c = 0; c < 2; ++c)
#pragma unroll
            for (int mm = 0; mm < 4; ++mm)
#pragma unroll
                for (int n = 0; n < 2; ++n) acc[a][c][mm][n] = (f32x4){0.f, 0.f, 0.f, 0.f};
    bf16x8 At[4][2], B0[2][2], B1[2][2];
    bool good = true;
    const char* fbase = ws + WS_FLAGA + (h * 2 + m) * 8192;
    unsigned fo = (unsigned)(lane & 31) * 4u, fva = 0u, fvb = 0u;
#define FLAG_ISSUE(fv, ds) do { if (wr == 0) asm volatile("global_load_dword %0, %1, %2 offset:%3 sc0 sc1" : "=v"(fv) : "v"(fo), "s"(fbase), "n"((ds) * 128) : "memory"); } while (0)
#define FLAG_CHECK(fv, ds) do { if (wr == 0 && t >= 2) { asm volatile("" : "+v"(fv)); \
        if (__builtin_amdgcn_ballot_w64((int)(fv - call) < 0) != 0ull) good = wait_flags((const unsigned*)(fbase + fo + (ds) * 128), call, tmo) && good; } } while (0)

    const char *a2, *b2, *a3, *b3; TILE_PTRS(2, a2, b2); TILE_PTRS(3, a3, b3);
    WAIT_V(10); BAR;
    if (wr == 1) BAR;
    STAMP(1);

    LDB(B0, 0, 0);
#define ITER(PUSH) do { \
        FLAG_ISSUE(fvb, 3); LDA(At, 0, 0); STA(1, 1, a1); \
        WAIT_V(10); WAIT_L(8); BAR; WAIT_LB(B0); MMAB(0, 0, At, B0); \
        if (PUSH) PUSH_SLOT(); LDB(B1, 0, 1); STB(0, 0, b2); \
        WAIT_V(10); BAR; WAIT_LB(B1); MMAB(0, 1, At, B1); \
        LDA(At, 0, 1); FLAG_CHECK(fva, 2); STA(0, 0, a2); \
        WAIT_V(10); BAR; WAIT_L(0); MMAB(1, 0, At, B0); \
        if (PUSH) PUSH_SLOT(); LDB(B0, 1, 0); STB(0, 1, b2); \
        WAIT_V(10); BAR; MMAB(1, 1, At, B1); \
        FLAG_ISSUE(fva, 4); LDA(At, 1, 0); STA(0, 1, a2); \
        WAIT_V(10); WAIT_L(8); BAR; WAIT_LB(B0); MMAB(0, 0, At, B0); \
        if (PUSH) PUSH_SLOT(); LDB(B1, 1, 1); STB(1, 0, b3); \
        WAIT_V(10); BAR; WAIT_LB(B1); MMAB(0, 1, At, B1); \
        LDA(At, 1, 1); FLAG_CHECK(fvb, 3); STA(1, 0, a3); \
        WAIT_V(10); BAR; WAIT_L(0); MMAB(1, 0, At, B0); \
        if (PUSH) PUSH_SLOT(); LDB(B0, 0, 0); STB(1, 1, b3); \
        a1 = a3; TILE_PTRS(t + 4, a2, b2); TILE_PTRS(t + 5, a3, b3); fo += 256u; SCHED; \
        WAIT_V(10); BAR; MMAB(1, 1, At, B1); \
        } while (0)

#pragma unroll 1
    for (int t = 0; t < TPUSH; t += 2) {
        if (t == 2) STAMP(2);
        if (t == 4) STAMP(3);
        if (t == 6) { STAMP(4); STAMPC(14); }
        if (t == 8) STAMP(5);
        ITER(1);
    }
#pragma unroll 1
    for (int t = TPUSH; t < NT - 2; t += 2) {
        if (t == 16) { STAMP(6); STAMPC(15); }
        if (t == 20) { STAMP(7); STAMPC(12); }
        ITER(0);
    }
    STAMP(8); STAMPC(13);

    int tz = threadIdx.x; asm volatile("" : "+v"(tz));
    const int lane2 = tz & 63, fr2 = lane2 & 15, fq2 = lane2 >> 4;
    {
        char* xs = ws + WS_XCH + ((size_t)b * 8 + wid) * 8192 + lane2 * 16;
#define XSEND(c) do { _Pragma("unroll") for (int mm = 0; mm < 4; ++mm) { \
            const f32x4 u0 = acc[0][c][mm][0], u1 = acc[0][c][mm][1]; \
            const u32x4 o = {cvt_pk_bf16(u0[0], u0[1]), cvt_pk_bf16(u0[2], u0[3]), cvt_pk_bf16(u1[0], u1[1]), cvt_pk_bf16(u1[2], u1[3])}; \
            asm volatile("global_store_dwordx4 %0, %1, off offset:%2 sc1\n\ts_nop 2" :: "v"(xs + (c) * 4096), "v"(o), "n"(mm * 1024) : "memory"); } } while (0)
        LDA(At, 0, 0); STA(1, 1, a1);
        WAIT_V(10); BAR; WAIT_LB(B0); MMAB(0, 0, At, B0);
        LDB(B1, 0, 1);
        WAIT_V(8); BAR; WAIT_LB(B1); MMAB(0, 1, At, B1);
        LDA(At, 0, 1);
        WAIT_V(6); BAR; WAIT_L(0); MMAB(1, 0, At, B0);
        LDB(B0, 1, 0);
        WAIT_V(4); BAR; MMAB(1, 1, At, B1);
        LDA(At, 1, 0);
        WAIT_V(2); BAR; WAIT_LB(B0); MMAB(0, 0, At, B0);
        LDB(B1, 1, 1); WAIT_V(0); XSEND(0); SCHED;
        BAR; WAIT_LB(B1); MMAB(0, 1, At, B1);
        LDA(At, 1, 1); XSEND(1); SCHED;
        BAR; WAIT_L(0); MMA(1, 0, At, B0); if (wr == 0) BAR;
        SCHED; MMA(1, 1, At, B1); SCHED;
        WAIT_V(0);
        if (lane2 == 0) __hip_atomic_store((unsigned*)(ws + WS_FLAGX + ((size_t)b * 8 + wid) * 128), call, __ATOMIC_RELAXED, __HIP_MEMORY_SCOPE_AGENT);
    }
    if (tz == 0) __hip_atomic_store(cnt, call, __ATOMIC_RELAXED, __HIP_MEMORY_SCOPE_AGENT);

    STAMP(9);
    {
        const int pb = b ^ 16;
        const unsigned* xf = (const unsigned*)(ws + WS_FLAGX + ((size_t)pb * 8 + wid) * 128);
        const unsigned long long t0 = __builtin_amdgcn_s_memrealtime();
        while ((int)((unsigned)__builtin_amdgcn_readfirstlane(__hip_atomic_load(xf, __ATOMIC_RELAXED, __HIP_MEMORY_SCOPE_AGENT)) - call) < 0) {
            if ((unsigned)__builtin_amdgcn_readfirstlane(__hip_atomic_load(tmo, __ATOMIC_RELAXED, __HIP_MEMORY_SCOPE_AGENT)) == call ||
                __builtin_amdgcn_s_memrealtime() - t0 > DEADLINE_TICKS) {
                __hip_atomic_store(tmo, call, __ATOMIC_RELAXED, __HIP_MEMORY_SCOPE_AGENT);
                break;
            }
            __builtin_amdgcn_s_sleep(1);
        }
        STAMP(10);
        const char* xr = ws + WS_XCH + ((size_t)pb * 8 + wid) * 8192 + lane2 * 16;
        u32x4 rv[2][4];
#pragma unroll
        for (int mm = 0; mm < 4; ++mm)
#pragma unroll
            for (int c = 0; c < 2; ++c)
                asm volatile("global_load_dwordx4 %0, %1, off sc1" : "=v"(rv[c][mm]) : "v"(xr + (c * 4 + mm) * 1024) : "memory");
        const bool bad = !good || (unsigned)__builtin_amdgcn_readfirstlane(__hip_atomic_load(tmo, __ATOMIC_RELAXED, __HIP_MEMORY_SCOPE_AGENT)) == call;
        WAIT_V(0);
#pragma unroll
        for (int c = 0; c < 2; ++c)
#pragma unroll
            for (int mm = 0; mm < 4; ++mm) asm volatile("" : "+v"(rv[c][mm]));
        char* op = out + (size_t)(m * 256 + h * 128 + wr * 64 + fr2) * (NCOL * 2) + (size_t)(ntile * 256 + wc * 32 + 8 * fq2) * 2;
        const bool odd = fq2 & 1;
#pragma unroll
        for (int mm = 0; mm < 4; ++mm)
#pragma unroll
            for (int c = 0; c < 2; ++c) {
                const u32x4 r = rv[c][mm];
                const f32x4 v0 = acc[1][c][mm][0] + (f32x4){__uint_as_float(r[0] << 16), __uint_as_float(r[0] & 0xFFFF0000u), __uint_as_float(r[1] << 16), __uint_as_float(r[1] & 0xFFFF0000u)};
                const f32x4 v1 = acc[1][c][mm][1] + (f32x4){__uint_as_float(r[2] << 16), __uint_as_float(r[2] & 0xFFFF0000u), __uint_as_float(r[3] << 16), __uint_as_float(r[3] & 0xFFFF0000u)};
                const unsigned l0 = cvt_pk_bf16(silu(v0[0]), silu(v0[1])), l1 = cvt_pk_bf16(silu(v0[2]), silu(v0[3]));
                const unsigned h0 = cvt_pk_bf16(silu(v1[0]), silu(v1[1])), h1 = cvt_pk_bf16(silu(v1[2]), silu(v1[3]));
                u32x4 o = odd ? (u32x4){h0, h1, l0, l1} : (u32x4){l0, l1, h0, h1};
                if (bad) o = (u32x4){0x7FC07FC0u, 0x7FC07FC0u, 0x7FC07FC0u, 0x7FC07FC0u};
                *(u32x4*)(op + (size_t)mm * 16 * (NCOL * 2) + c * 256) = o;
            }
        STAMP(11);
    }
}

extern "C" void kernel_launch(int rank, int nranks, ncclComm_t comm,
                              hipStream_t stream,
                              void* const* d_in, const int* in_sizes, int n_in,
                              void* d_out, int out_size,
                              const int* in_dtypes, int out_dtype,
                              void* const* d_out_all,
                              void* const* d_ws_all, size_t ws_bytes) {
    if (nranks != NR || (unsigned)rank >= (unsigned)NR || ws_bytes < WS_END) { fprintf(stderr, "unsupported configuration\n"); return; }
    static bool init = false;
    if (!init) {
        HIP_CHECK(hipFuncSetAttribute((const void*)a2a_gemm_silu, hipFuncAttributeMaxDynamicSharedMemorySize, LDS_BYTES));
        init = true;
    }
    hipLaunchKernelGGL(a2a_gemm_silu, dim3(NWG), dim3(512), LDS_BYTES, stream,
                       (const char*)d_in[0], (const char*)d_in[1], (char*)d_out, d_ws_all, rank);
    HIP_CHECK(hipGetLastError());
}
```

```cpp
#include <hip/hip_runtime.h>
#include <rccl/rccl.h>
#include <cstdio>

#define HIP_CHECK(expr) do { hipError_t _e = (expr); if (_e != hipSuccess) { \
    fprintf(stderr, "HIP error %s at %s:%d\n", hipGetErrorString(_e), __FILE__, __LINE__); \
    return; } } while (0)

#define LAS __attribute__((address_space(3)))
typedef short bf16x8 __attribute__((ext_vector_type(8)));
typedef short s16x4 __attribute__((ext_vector_type(4)));
typedef float f32x4 __attribute__((ext_vector_type(4)));
typedef unsigned u32x4 __attribute__((ext_vector_type(4)));

constexpr int NR = 8;
constexpr int NCOL = 16384;
constexpr int KCH = 512;
constexpr int HTB = 128 * 64 * 2;
constexpr int LDS_BYTES = 8 * HTB;
constexpr int NWG = 256;
constexpr int TPUSH = 10;
#define COFF(c) (((c) >> 1) * 3)
constexpr int NT = 32;

constexpr size_t WS_TMO   = 0;
constexpr size_t WS_CNT   = 4096;
constexpr size_t WS_FLAGA = WS_CNT + NWG * 128;
constexpr size_t WS_FLAGX = WS_FLAGA + 32768;
constexpr size_t WS_SYNC  = WS_FLAGX + NWG * 8 * 128;
constexpr size_t WS_RECV  = 1u << 20;
constexpr size_t BLK_BYTES = (size_t)KCH * KCH * 2;
constexpr size_t WS_XCH   = WS_RECV + 2 * NR * BLK_BYTES;
constexpr size_t WS_END   = WS_XCH + (size_t)NWG * 8 * 16 * 1024;
constexpr unsigned long long DEADLINE_TICKS = 40000000ull;

__device__ __forceinline__ int lds_byte(int r, int c) { const int st = (r >> 4) * 2 + (c >> 5), rr = r & 15, cc = c & 31, ob = rr * 64 + cc * 2; return st * 1024 + (ob ^ (((ob >> 9) & 1) << 5)); }
__device__ __forceinline__ void stage_rc(int b, int& R, int& C) { const int st = b / 1024, sb = b % 1024, swz = sb ^ (((sb >> 9) & 1) << 5); R = (st >> 1) * 16 + swz / 64; C = (st & 1) * 32 + (swz % 64) / 2; }
__device__ __forceinline__ unsigned cvt_pk_bf16(float lo, float hi) { unsigned r; asm volatile("v_cvt_pk_bf16_f32 %0, %1, %2" : "=v"(r) : "v"(lo), "v"(hi)); return r; }
__device__ __forceinline__ float silu(float y) {
    const float c = __builtin_amdgcn_fmed3f(y, -60.0f, 60.0f);
    const float e = __builtin_amdgcn_exp2f(c * -1.44269504088896f);
    return y * __builtin_amdgcn_rcpf(1.0f + e);
}

#define STAMP(k) do {} while (0)
#define STAMPC(k) do {} while (0)
#define WAIT_V(n) asm volatile("s_waitcnt vmcnt(" #n ")" ::: "memory")
#define WAIT_L(n) asm volatile("s_waitcnt lgkmcnt(" #n ")" ::: "memory")
#define BAR __builtin_amdgcn_s_barrier()
#define SCHED __builtin_amdgcn_sched_barrier(0)

__device__ __forceinline__ bool wait_flags(const unsigned* f, unsigned val, unsigned call, unsigned* tmo) {
    const unsigned long long t0 = __builtin_amdgcn_s_memrealtime();
    for (;;) {
        const unsigned v = __hip_atomic_load(f, __ATOMIC_RELAXED, __HIP_MEMORY_SCOPE_SYSTEM);
        const bool ok = (int)(v - val) >= 0;
        if (__builtin_amdgcn_ballot_w64(!ok) == 0ull) return true;
        if ((unsigned)__builtin_amdgcn_readfirstlane(__hip_atomic_load(tmo, __ATOMIC_RELAXED, __HIP_MEMORY_SCOPE_AGENT)) == call ||
            __builtin_amdgcn_s_memrealtime() - t0 > DEADLINE_TICKS) {
            __hip_atomic_store(tmo, call, __ATOMIC_RELAXED, __HIP_MEMORY_SCOPE_AGENT);
            return false;
        }
        __builtin_amdgcn_s_sleep(1);
    }
}

__global__ __launch_bounds__(512, 2)
void a2a_gemm_silu(const char* __restrict__ x, const char* __restrict__ w, char* __restrict__ out,
                   void* const* __restrict__ ws_all, int rank) {
    extern __shared__ __attribute__((aligned(16))) unsigned char shm[];
    LAS unsigned char* lds = (LAS unsigned char*)shm;
    const int tid = threadIdx.x, wid = __builtin_amdgcn_readfirstlane(tid >> 6), lane = tid & 63;
    const int wr = wid >> 2, wc = wid & 3, fr = lane & 15, fq = lane >> 4;
    const int b = blockIdx.x, idx = b >> 3;
    const int ntile = (b & 7) * 8 + (idx >> 2), m = idx & 1, h = (idx >> 1) & 1;
    char* ws = (char*)ws_all[rank];
    unsigned* tmo = (unsigned*)(ws + WS_TMO);
    STAMP(0);

    unsigned* cnt = (unsigned*)(ws + WS_CNT + (size_t)b * 128);
    unsigned cntv;
    asm volatile("global_load_dword %0, %1, off sc1" : "=v"(cntv) : "v"(cnt) : "memory");

    unsigned voffA[2], voffB[2];
#pragma unroll
    for (int i = 0; i < 2; ++i) {
        int R, C; stage_rc(tid * 16 + i * 8192, R, C);
        voffA[i] = (unsigned)(R * KCH + C) * 2u;
        const int r = wid * 4 + (lane >> 4) + i * 32, c = (lane & 15) ^ (((r & 3) << 2) | ((r >> 3) & 1));
        voffB[i] = (unsigned)r * (unsigned)(NCOL * 2) + (unsigned)c * 16u;
    }
    const unsigned ldsw = (unsigned)wid * 1024u;
    const int aoff = lds_byte(wr * 64 + fr, fq * 8);
    const int q4 = (lane & 15) >> 2, p4 = lane & 3;
    const int boff0 = 256 * (8 * fq + q4) + 16 * ((4 * wc + p4) ^ (q4 << 2) ^ (fq & 1)) + 8 * (p4 & 1);
    const unsigned bad0 = (unsigned)(size_t)lds + 65536u + (unsigned)boff0, bad1 = bad0 ^ 8u;

    const int flip = h ^ 1;
    const long aoffs0 = (long)(m * 256 + flip * 128) * (KCH * 2) + h * 512;
    const long hstepA = (flip ? -1L : 1L) * 128 * (KCH * 2);
    const char* xloc = x + (size_t)rank * BLK_BYTES + aoffs0;
    const char* wbase = w + (size_t)(h * 256) * (NCOL * 2) + (size_t)ntile * 512;
    constexpr long hstepB = 256;

    const int coff = COFF(h * 2 + m);
#define TILE_PTRS(s_, A_, B_) do { const int _s = (s_) & 31; const int _u = _s < 4 ? 0 : _s - 4; const int _j0 = (_u * 37) >> 8; \
        const int _d0 = _u - _j0 * 7 + coff; const int _d = _d0 >= 7 ? _d0 - 7 : _d0; \
        const int _j = _s < 4 ? _s : _j0; const int _p = _s < 4 ? rank : ((rank + 1 + _d) & 7); \
        A_ = _s < 4 ? xloc + _j * 128 : xrem + (size_t)_j * BLK_BYTES + _p * 128; \
        B_ = wbase + (size_t)(_p * KCH + _j * 64) * (NCOL * 2); } while (0)

#define SA(bb, hh) (((bb) * 2 + (hh)) * HTB)
#define SB(bb, hh) ((4 + (bb) * 2 + (hh)) * HTB)
#define WAUX 2
#define GLDS(gp, loff, aux) __builtin_amdgcn_global_load_lds((const unsigned*)(gp), (LAS unsigned*)(lds + (loff)), 16, 0, aux)
#define STAGE(bufoff, gbase, voff, aux) do { GLDS((gbase) + (voff)[0], (bufoff) + ldsw, aux); GLDS((gbase) + (voff)[1], (bufoff) + ldsw + 8192, aux); } while (0)
#define STA(bb, hh, ap) STAGE(SA(bb, hh), (ap) + (hh) * hstepA, voffA, 0)
#define STB(bb, hh, bp) STAGE(SB(bb, hh), (bp) + (hh) * hstepB, voffB, WAUX)
#define LDA(dst, bb, hh) do { _Pragma("unroll") for (int mm = 0; mm < 4; ++mm) _Pragma("unroll") for (int k = 0; k < 2; ++k) \
        dst[mm][k] = *(const LAS bf16x8*)(lds + SA(bb, hh) + aoff + mm * 2048 + k * 1024); } while (0)
#define TRD(dst, base, off) asm volatile("ds_read_b64_tr_b16 %0, %1 offset:%2" : "=v"(dst) : "v"(base), "n"(off) : "memory")
#define LDB(dst, bb, hh) do { _Pragma("unroll") for (int n = 0; n < 2; ++n) _Pragma("unroll") for (int k = 0; k < 2; ++k) { \
        s16x4 _lo, _hi; TRD(_lo, (n ? bad1 : bad0), SB(bb, hh) - 65536 + k * 8192); TRD(_hi, (n ? bad1 : bad0), SB(bb, hh) - 65536 + k * 8192 + 1024); \
        dst[n][k] = __builtin_shufflevector(_lo, _hi, 0, 1, 2, 3, 4, 5, 6, 7); } } while (0)
#define WAIT_LB(Bt_) asm volatile("s_waitcnt lgkmcnt(0)" : "+v"(Bt_[0][0]), "+v"(Bt_[0][1]), "+v"(Bt_[1][0]), "+v"(Bt_[1][1]) :: "memory")
#define MMA(ai, bj, At_, Bt_) do { __builtin_amdgcn_s_setprio(1); _Pragma("unroll") for (int mm = 0; mm < 4; ++mm) _Pragma("unroll") for (int n = 0; n < 2; ++n) _Pragma("unroll") for (int k = 0; k < 2; ++k) \
        acc[ai][bj][mm][n] = __builtin_amdgcn_mfma_f32_16x16x32_bf16(Bt_[n][k], At_[mm][k], acc[ai][bj][mm][n], 0, 0, 0); __builtin_amdgcn_s_setprio(0); } while (0)

#define EARLYB 0
#define MMAB(ai, bj, At_, Bt_) do { __builtin_amdgcn_s_setprio(1); _Pragma("unroll") for (int mm = 0; mm < 4; ++mm) { \
        if (mm == 4 - EARLYB) { SCHED; BAR; SCHED; } \
        _Pragma("unroll") for (int n = 0; n < 2; ++n) _Pragma("unroll") for (int k = 0; k < 2; ++k) \
        acc[ai][bj][mm][n] = __builtin_amdgcn_mfma_f32_16x16x32_bf16(Bt_[n][k], At_[mm][k], acc[ai][bj][mm][n], 0, 0, 0); } \
        __builtin_amdgcn_s_setprio(0); if (EARLYB == 0) BAR; SCHED; } while (0)

    const char *a0 = xloc, *b0 = wbase + (size_t)(rank * KCH) * (NCOL * 2), *a1 = a0 + 128, *b1 = b0 + (size_t)64 * (NCOL * 2);
    const bool pusher = b < 224;
    const int k1 = (b >> 2) & 7;
    const unsigned dst = (unsigned)(rank + 1 + (b >> 5)) & 7u;
    int pcl = 0, uc = 0;
    {
        int u[4];
#pragma unroll
        for (int c = 0; c < 4; ++c) { const int v = 6 - (b >> 5) - COFF(c); u[c] = v < 0 ? v + 7 : v; }
#pragma unroll
        for (int c = 0; c < 4; ++c) {
            const int o = (u[0] * 4 + 0 < u[c] * 4 + c) + (u[1] * 4 + 1 < u[c] * 4 + c) + (u[2] * 4 + 2 < u[c] * 4 + c) + (u[3] * 4 + 3 < u[c] * 4 + c);
            if (o == (k1 & 3)) { pcl = c; uc = u[c]; }
        }
    }
    const int pj = k1 >> 2;
    const unsigned loff = (unsigned)((pcl & 1) * 256 + (b & 3) * 64 + wid * 8 + (lane >> 3)) * (KCH * 2) + (lane & 7) * 16;
    const char* psrc = x + (size_t)dst * BLK_BYTES + (pcl >> 1) * 512 + pj * 128;
    u32x4 pv;
    if (pusher) asm volatile("global_load_dwordx4 %0, %1, %2 nt" : "=v"(pv) : "v"(loff), "s"(psrc) : "memory");
    STB(0, 0, b0); STA(0, 0, a0); STB(0, 1, b0); STA(0, 1, a0); STB(1, 0, b1); STA(1, 0, a1); STB(1, 1, b1);
    char* pws = (char*)ws_all[dst];
    WAIT_V(14);
    asm volatile("" : "+v"(cntv));
    const unsigned call = (unsigned)__builtin_amdgcn_readfirstlane(cntv) + 1u;
    const unsigned par = call & 1u;
    const char* xrem = ws + WS_RECV + (size_t)(par * 8 + h * 4) * BLK_BYTES + (size_t)(m * 256 + flip * 128) * (KCH * 2);
    char* pdst = pws + WS_RECV + (size_t)(par * 8 + (pcl >> 1) * 4 + pj) * BLK_BYTES + rank * 128;
    char* pflag = pws + WS_FLAGA + (size_t)(pcl * 64 + 4 + 7 * pj + uc) * 128 + ((b & 3) * 8 + wid) * 4;
    int ps = 0, ev = pusher ? k1 + 1 : 1000;
#define PUSH_SLOT() do { if (--ev == 0) { \
        if (ps & 1) { WAIT_V(0); if (lane == 0) asm volatile("global_store_dword %0, %1, %2 sc0 sc1" :: "v"(0u), "v"(call), "s"(pflag) : "memory"); pflag += 14 * 128; } \
        else { asm volatile("global_store_dwordx4 %0, %1, %2 sc0 sc1\n\ts_nop 2" :: "v"(loff), "v"(pv), "s"(pdst) : "memory"); pdst += 2 * BLK_BYTES; \
               if (ps == 0) asm volatile("global_load_dwordx4 %0, %1, %2 offset:256 nt" : "=v"(pv) : "v"(loff), "s"(psrc) : "memory"); } \
        ++ps; ev = ps < 4 ? 4 : 1000; } } while (0)
    PUSH_SLOT();

    f32x4 acc[2][2][4][2];
#pragma unroll
    for (int a = 0; a < 2; ++a)
#pragma unroll
        for (int c = 0; c < 2; ++c)
#pragma unroll
            for (int mm = 0; mm < 4; ++mm)
#pragma unroll
                for (int n = 0; n < 2; ++n) acc[a][c][mm][n] = (f32x4){0.f, 0.f, 0.f, 0.f};
    bf16x8 At[4][2], B0[2][2], B1[2][2];
    bool good = true;
    const char* fbase = ws + WS_FLAGA + (h * 2 + m) * 8192;
    unsigned fo = (unsigned)(lane & 31) * 4u, fva = 0u, fvb = 0u;
#define FLAG_ISSUE(fv, ds) do { if (wr == 0) asm volatile("global_load_dword %0, %1, %2 offset:%3 sc0 sc1" : "=v"(fv) : "v"(fo), "s"(fbase), "n"((ds) * 128) : "memory"); } while (0)
#define FLAG_CHECK(fv, ds) do { if (wr == 0 && t >= 2) { asm volatile("" : "+v"(fv)); \
        if (__builtin_amdgcn_ballot_w64((int)(fv - call) < 0) != 0ull) good = wait_flags((const unsigned*)(fbase + fo + (ds) * 128), call, call, tmo) && good; } } while (0)

    const char *a2, *b2, *a3, *b3; TILE_PTRS(2, a2, b2); TILE_PTRS(3, a3, b3);
    const char* sbase = ws + WS_SYNC;
    unsigned sv = 0u, sval = call * 16u;
#define PAIR_POST() do { ++sval; if (wid == 0 && lane == 0) asm volatile("global_store_dword %0, %1, %2 sc1" :: "v"(b * 128), "v"(sval), "s"(sbase) : "memory"); } while (0)
#define PAIR_ISSUE() do { if (wid == 0) asm volatile("global_load_dword %0, %1, %2 sc1" : "=v"(sv) : "v"((b ^ 8) * 128), "s"(sbase) : "memory"); } while (0)
#define PAIR_CHECK() do { if (wid == 0 && t >= 2) { asm volatile("" : "+v"(sv)); \
        if ((int)((unsigned)__builtin_amdgcn_readfirstlane(sv) - (sval - 1u)) < 0) good = wait_flags((const unsigned*)(sbase + (b ^ 8) * 128), sval, call, tmo) && good; } } while (0)
    WAIT_V(10); BAR;
    if (wr == 1) BAR;
    STAMP(1);

    LDB(B0, 0, 0);
#define ITER(PUSH) do { \
        PAIR_POST(); FLAG_ISSUE(fvb, 3); LDA(At, 0, 0); STA(1, 1, a1); \
        WAIT_V(10); WAIT_L(8); BAR; WAIT_LB(B0); MMAB(0, 0, At, B0); \
        if (PUSH) PUSH_SLOT(); LDB(B1, 0, 1); STB(0, 0, b2); \
        WAIT_V(10); BAR; WAIT_LB(B1); MMAB(0, 1, At, B1); \
        LDA(At, 0, 1); FLAG_CHECK(fva, 2); STA(0, 0, a2); \
        WAIT_V(10); BAR; WAIT_L(0); MMAB(1, 0, At, B0); \
        if (PUSH) PUSH_SLOT(); LDB(B0, 1, 0); STB(0, 1, b2); \
        WAIT_V(10); BAR; MMAB(1, 1, At, B1); \
        PAIR_CHECK(); FLAG_ISSUE(fva, 4); LDA(At, 1, 0); STA(0, 1, a2); \
        WAIT_V(10); WAIT_L(8); BAR; WAIT_LB(B0); MMAB(0, 0, At, B0); \
        if (PUSH) PUSH_SLOT(); LDB(B1, 1, 1); STB(1, 0, b3); \
        WAIT_V(10); BAR; WAIT_LB(B1); MMAB(0, 1, At, B1); \
        LDA(At, 1, 1); FLAG_CHECK(fvb, 3); PAIR_ISSUE(); STA(1, 0, a3); \
        WAIT_V(10); BAR; WAIT_L(0); MMAB(1, 0, At, B0); \
        if (PUSH) PUSH_SLOT(); LDB(B0, 0, 0); STB(1, 1, b3); \
        a1 = a3; TILE_PTRS(t + 4, a2, b2); TILE_PTRS(t + 5, a3, b3); fo += 256u; SCHED; \
        WAIT_V(10); BAR; MMAB(1, 1, At, B1); \
        } while (0)

#pragma unroll 1
    for (int t = 0; t < TPUSH; t += 2) {
        if (t == 2) STAMP(2);
        if (t == 4) STAMP(3);
        if (t == 6) { STAMP(4); STAMPC(14); }
        if (t == 8) STAMP(5);
        ITER(1);
    }
#pragma unroll 1
    for (int t = TPUSH; t < NT - 2; t += 2) {
        if (t == 16) { STAMP(6); STAMPC(15); }
        if (t == 20) { STAMP(7); STAMPC(12); }
        ITER(0);
    }
    STAMP(8); STAMPC(13);

    int tz = threadIdx.x; asm volatile("" : "+v"(tz));
    const int lane2 = tz & 63, fr2 = lane2 & 15, fq2 = lane2 >> 4;
    {
        char* xs = ws + WS_XCH + ((size_t)b * 8 + wid) * 8192 + lane2 * 16;
#define XSEND(c) do { _Pragma("unroll") for (int mm = 0; mm < 4; ++mm) { \
            const f32x4 u0 = acc[0][c][mm][0], u1 = acc[0][c][mm][1]; \
            const u32x4 o = {cvt_pk_bf16(u0[0], u0[1]), cvt_pk_bf16(u0[2], u0[3]), cvt_pk_bf16(u1[0], u1[1]), cvt_pk_bf16(u1[2], u1[3])}; \
            asm volatile("global_store_dwordx4 %0, %1, off offset:%2 sc1\n\ts_nop 2" :: "v"(xs + (c) * 4096), "v"(o), "n"(mm * 1024) : "memory"); } } while (0)
        LDA(At, 0, 0); STA(1, 1, a1);
        WAIT_V(10); BAR; WAIT_LB(B0); MMAB(0, 0, At, B0);
        LDB(B1, 0, 1);
        WAIT_V(8); BAR; WAIT_LB(B1); MMAB(0, 1, At, B1);
        LDA(At, 0, 1);
        WAIT_V(6); BAR; WAIT_L(0); MMAB(1, 0, At, B0);
        LDB(B0, 1, 0);
        WAIT_V(4); BAR; MMAB(1, 1, At, B1);
        LDA(At, 1, 0);
        WAIT_V(2); BAR; WAIT_LB(B0); MMAB(0, 0, At, B0);
        LDB(B1, 1, 1); WAIT_V(0); XSEND(0); SCHED;
        BAR; WAIT_LB(B1); MMAB(0, 1, At, B1);
        LDA(At, 1, 1); XSEND(1); SCHED;
        BAR; WAIT_L(0); MMA(1, 0, At, B0); if (wr == 0) BAR;
        SCHED; MMA(1, 1, At, B1); SCHED;
        WAIT_V(0);
        if (lane2 == 0) __hip_atomic_store((unsigned*)(ws + WS_FLAGX + ((size_t)b * 8 + wid) * 128), call, __ATOMIC_RELAXED, __HIP_MEMORY_SCOPE_AGENT);
    }
    if (tz == 0) __hip_atomic_store(cnt, call, __ATOMIC_RELAXED, __HIP_MEMORY_SCOPE_AGENT);

    STAMP(9);
    {
        const int pb = b ^ 16;
        const unsigned* xf = (const unsigned*)(ws + WS_FLAGX + ((size_t)pb * 8 + wid) * 128);
        const unsigned long long t0 = __builtin_amdgcn_s_memrealtime();
        while ((int)((unsigned)__builtin_amdgcn_readfirstlane(__hip_atomic_load(xf, __ATOMIC_RELAXED, __HIP_MEMORY_SCOPE_AGENT)) - call) < 0) {
            if ((unsigned)__builtin_amdgcn_readfirstlane(__hip_atomic_load(tmo, __ATOMIC_RELAXED, __HIP_MEMORY_SCOPE_AGENT)) == call ||
                __builtin_amdgcn_s_memrealtime() - t0 > DEADLINE_TICKS) {
                __hip_atomic_store(tmo, call, __ATOMIC_RELAXED, __HIP_MEMORY_SCOPE_AGENT);
                break;
            }
            __builtin_amdgcn_s_sleep(1);
        }
        STAMP(10);
        const char* xr = ws + WS_XCH + ((size_t)pb * 8 + wid) * 8192 + lane2 * 16;
        u32x4 rv[2][4];
#pragma unroll
        for (int mm = 0; mm < 4; ++mm)
#pragma unroll
            for (int c = 0; c < 2; ++c)
                asm volatile("global_load_dwordx4 %0, %1, off sc1" : "=v"(rv[c][mm]) : "v"(xr + (c * 4 + mm) * 1024) : "memory");
        const bool bad = !good || (unsigned)__builtin_amdgcn_readfirstlane(__hip_atomic_load(tmo, __ATOMIC_RELAXED, __HIP_MEMORY_SCOPE_AGENT)) == call;
        WAIT_V(0);
#pragma unroll
        for (int c = 0; c < 2; ++c)
#pragma unroll
            for (int mm = 0; mm < 4; ++mm) asm volatile("" : "+v"(rv[c][mm]));
        char* op = out + (size_t)(m * 256 + h * 128 + wr * 64 + fr2) * (NCOL * 2) + (size_t)(ntile * 256 + wc * 32 + 8 * fq2) * 2;
        const bool odd = fq2 & 1;
#pragma unroll
        for (int mm = 0; mm < 4; ++mm)
#pragma unroll
            for (int c = 0; c < 2; ++c) {
                const u32x4 r = rv[c][mm];
                const f32x4 v0 = acc[1][c][mm][0] + (f32x4){__uint_as_float(r[0] << 16), __uint_as_float(r[0] & 0xFFFF0000u), __uint_as_float(r[1] << 16), __uint_as_float(r[1] & 0xFFFF0000u)};
                const f32x4 v1 = acc[1][c][mm][1] + (f32x4){__uint_as_float(r[2] << 16), __uint_as_float(r[2] & 0xFFFF0000u), __uint_as_float(r[3] << 16), __uint_as_float(r[3] & 0xFFFF0000u)};
                const unsigned l0 = cvt_pk_bf16(silu(v0[0]), silu(v0[1])), l1 = cvt_pk_bf16(silu(v0[2]), silu(v0[3]));
                const unsigned h0 = cvt_pk_bf16(silu(v1[0]), silu(v1[1])), h1 = cvt_pk_bf16(silu(v1[2]), silu(v1[3]));
                u32x4 o = odd ? (u32x4){h0, h1, l0, l1} : (u32x4){l0, l1, h0, h1};
                if (bad) o = (u32x4){0x7FC07FC0u, 0x7FC07FC0u, 0x7FC07FC0u, 0x7FC07FC0u};
                *(u32x4*)(op + (size_t)mm * 16 * (NCOL * 2) + c * 256) = o;
            }
        STAMP(11);
    }
}

extern "C" void kernel_launch(int rank, int nranks, ncclComm_t comm,
                              hipStream_t stream,
                              void* const* d_in, const int* in_sizes, int n_in,
                              void* d_out, int out_size,
                              const int* in_dtypes, int out_dtype,
                              void* const* d_out_all,
                              void* const* d_ws_all, size_t ws_bytes) {
    if (nranks != NR || (unsigned)rank >= (unsigned)NR || ws_bytes < WS_END) { fprintf(stderr, "unsupported configuration\n"); return; }
    static bool init = false;
    if (!init) {
        HIP_CHECK(hipFuncSetAttribute((const void*)a2a_gemm_silu, hipFuncAttributeMaxDynamicSharedMemorySize, LDS_BYTES));
        init = true;
    }
    hipLaunchKernelGGL(a2a_gemm_silu, dim3(NWG), dim3(512), LDS_BYTES, stream,
                       (const char*)d_in[0], (const char*)d_in[1], (char*)d_out, d_ws_all, rank);
    HIP_CHECK(hipGetLastError());
}
```

```cpp
#include <hip/hip_runtime.h>
#include <rccl/rccl.h>
#include <cstdio>

#define HIP_CHECK(expr) do { hipError_t _e = (expr); if (_e != hipSuccess) { \
    fprintf(stderr, "HIP error %s at %s:%d\n", hipGetErrorString(_e), __FILE__, __LINE__); \
    return; } } while (0)

#define LAS __attribute__((address_space(3)))
typedef short bf16x8 __attribute__((ext_vector_type(8)));
typedef short s16x4 __attribute__((ext_vector_type(4)));
typedef float f32x4 __attribute__((ext_vector_type(4)));
typedef unsigned u32x4 __attribute__((ext_vector_type(4)));

constexpr int NR = 8;
constexpr int NCOL = 16384;
constexpr int KCH = 512;
constexpr int HTB = 128 * 64 * 2;
constexpr int LDS_DEC = 8 * HTB;
constexpr int LDS_BYTES = LDS_DEC + 64;
constexpr int NWG = 256;
constexpr int TPUSH = 10;
#define PAT 150ull
#define LATEALL 1
#define PFMAX 0
#define PSL 2u
#define GAP1 2
#define COFF(c) (((c) >> 1) * 3)
constexpr int NT = 32;

constexpr size_t WS_TMO   = 0;
constexpr size_t WS_CNT   = 4096;
constexpr size_t WS_FLAGA = WS_CNT + NWG * 128;
constexpr size_t WS_FLAGX = WS_FLAGA + 32768;
constexpr size_t WS_SYNC  = WS_FLAGX + NWG * 8 * 128;
constexpr size_t WS_RECV  = 1u << 20;
constexpr size_t BLK_BYTES = (size_t)KCH * KCH * 2;
constexpr size_t WS_XCH   = WS_RECV + 2 * NR * BLK_BYTES;
constexpr size_t WS_END   = WS_XCH + (size_t)NWG * 8 * 16 * 1024;
constexpr unsigned long long DEADLINE_TICKS = 40000000ull;

__device__ __forceinline__ int lds_byte(int r, int c) { const int st = (r >> 4) * 2 + (c >> 5), rr = r & 15, cc = c & 31, ob = rr * 64 + cc * 2; return st * 1024 + (ob ^ (((ob >> 9) & 1) << 5)); }
__device__ __forceinline__ void stage_rc(int b, int& R, int& C) { const int st = b / 1024, sb = b % 1024, swz = sb ^ (((sb >> 9) & 1) << 5); R = (st >> 1) * 16 + swz / 64; C = (st & 1) * 32 + (swz % 64) / 2; }
__device__ __forceinline__ unsigned cvt_pk_bf16(float lo, float hi) { unsigned r; asm volatile("v_cvt_pk_bf16_f32 %0, %1, %2" : "=v"(r) : "v"(lo), "v"(hi)); return r; }
__device__ __forceinline__ float silu(float y) {
    const float c = __builtin_amdgcn_fmed3f(y, -60.0f, 60.0f);
    const float e = __builtin_amdgcn_exp2f(c * -1.44269504088896f);
    return y * __builtin_amdgcn_rcpf(1.0f + e);
}

#define STAMP(k) do {} while (0)
#define STAMPC(k) do {} while (0)
#define WAIT_V(n) asm volatile("s_waitcnt vmcnt(" #n ")" ::: "memory")
#define WAIT_L(n) asm volatile("s_waitcnt lgkmcnt(" #n ")" ::: "memory")
#define BAR __builtin_amdgcn_s_barrier()
#define SCHED __builtin_amdgcn_sched_barrier(0)

__device__ __forceinline__ bool wait_flags(const unsigned* f, unsigned val, unsigned call, unsigned* tmo) {
    const unsigned long long t0 = __builtin_amdgcn_s_memrealtime();
    for (;;) {
        const unsigned v = __hip_atomic_load(f, __ATOMIC_RELAXED, __HIP_MEMORY_SCOPE_SYSTEM);
        const bool ok = (int)(v - val) >= 0;
        if (__builtin_amdgcn_ballot_w64(!ok) == 0ull) return true;
        if ((unsigned)__builtin_amdgcn_readfirstlane(__hip_atomic_load(tmo, __ATOMIC_RELAXED, __HIP_MEMORY_SCOPE_AGENT)) == call ||
            __builtin_amdgcn_s_memrealtime() - t0 > DEADLINE_TICKS) {
            __hip_atomic_store(tmo, call, __ATOMIC_RELAXED, __HIP_MEMORY_SCOPE_AGENT);
            return false;
        }
        __builtin_amdgcn_s_sleep(1);
    }
}

__global__ __launch_bounds__(512, 2)
void a2a_gemm_silu(const char* __restrict__ x, const char* __restrict__ w, char* __restrict__ out,
                   void* const* __restrict__ ws_all, int rank) {
    extern __shared__ __attribute__((aligned(16))) unsigned char shm[];
    LAS unsigned char* lds = (LAS unsigned char*)shm;
    const int tid = threadIdx.x, wid = __builtin_amdgcn_readfirstlane(tid >> 6), lane = tid & 63;
    const int wr = wid >> 2, wc = wid & 3, fr = lane & 15, fq = lane >> 4;
    const int b = blockIdx.x, idx = b >> 3;
    const int ntile = (b & 7) * 8 + (idx >> 2), m = idx & 1, h = (idx >> 1) & 1;
    char* ws = (char*)ws_all[rank];
    unsigned* tmo = (unsigned*)(ws + WS_TMO);
    STAMP(0);

    unsigned* cnt = (unsigned*)(ws + WS_CNT + (size_t)b * 128);
    unsigned cntv;
    asm volatile("global_load_dword %0, %1, off sc1" : "=v"(cntv) : "v"(cnt) : "memory");

    unsigned voffA[2], voffB[2];
#pragma unroll
    for (int i = 0; i < 2; ++i) {
        int R, C; stage_rc(tid * 16 + i * 8192, R, C);
        voffA[i] = (unsigned)(R * KCH + C) * 2u;
        const int r = wid * 4 + (lane >> 4) + i * 32, c = (lane & 15) ^ (((r & 3) << 2) | ((r >> 3) & 1));
        voffB[i] = (unsigned)r * (unsigned)(NCOL * 2) + (unsigned)c * 16u;
    }
    const unsigned ldsw = (unsigned)wid * 1024u;
    const int aoff = lds_byte(wr * 64 + fr, fq * 8);
    const int q4 = (lane & 15) >> 2, p4 = lane & 3;
    const int boff0 = 256 * (8 * fq + q4) + 16 * ((4 * wc + p4) ^ (q4 << 2) ^ (fq & 1)) + 8 * (p4 & 1);
    const unsigned bad0 = (unsigned)(size_t)lds + 65536u + (unsigned)boff0, bad1 = bad0 ^ 8u;

    const int flip = h ^ 1;
    const long aoffs0 = (long)(m * 256 + flip * 128) * (KCH * 2) + h * 512;
    const long hstepA = (flip ? -1L : 1L) * 128 * (KCH * 2);
    const char* xloc = x + (size_t)rank * BLK_BYTES + aoffs0;
    const char* wbase = w + (size_t)(h * 256) * (NCOL * 2) + (size_t)ntile * 512;
    constexpr long hstepB = 256;

    const int coff = COFF(h * 2 + m);
#define TILE_PTRS(u_, A_, B_) do { const int _u = (u_); const int _j = (_u * 37) >> 8; \
        const int _d0 = _u - _j * 7 + coff; const int _d = _d0 >= 7 ? _d0 - 7 : _d0; const int _p = (rank + 1 + _d) & 7; \
        A_ = xrem + (size_t)_j * BLK_BYTES + _p * 128; \
        B_ = wbase + (size_t)(_p * KCH + _j * 64) * (NCOL * 2); } while (0)

#define SA(bb, hh) (((bb) * 2 + (hh)) * HTB)
#define SB(bb, hh) ((4 + (bb) * 2 + (hh)) * HTB)
#define OUTNT 0
#define WAUX 2
#define GLDS(gp, loff, aux) __builtin_amdgcn_global_load_lds((const unsigned*)(gp), (LAS unsigned*)(lds + (loff)), 16, 0, aux)
#define STAGE(bufoff, gbase, voff, aux) do { GLDS((gbase) + (voff)[0], (bufoff) + ldsw, aux); GLDS((gbase) + (voff)[1], (bufoff) + ldsw + 8192, aux); } while (0)
#define STA(bb, hh, ap) STAGE(SA(bb, hh), (ap) + (hh) * hstepA, voffA, 0)
#define STB(bb, hh, bp) STAGE(SB(bb, hh), (bp) + (hh) * hstepB, voffB, WAUX)
#define LDA(dst, bb, hh) do { _Pragma("unroll") for (int mm = 0; mm < 4; ++mm) _Pragma("unroll") for (int k = 0; k < 2; ++k) \
        dst[mm][k] = *(const LAS bf16x8*)(lds + SA(bb, hh) + aoff + mm * 2048 + k * 1024); } while (0)
#define TRD(dst, base, off) asm volatile("ds_read_b64_tr_b16 %0, %1 offset:%2" : "=v"(dst) : "v"(base), "n"(off) : "memory")
#define LDB(dst, bb, hh) do { _Pragma("unroll") for (int n = 0; n < 2; ++n) _Pragma("unroll") for (int k = 0; k < 2; ++k) { \
        s16x4 _lo, _hi; TRD(_lo, (n ? bad1 : bad0), SB(bb, hh) - 65536 + k * 8192); TRD(_hi, (n ? bad1 : bad0), SB(bb, hh) - 65536 + k * 8192 + 1024); \
        dst[n][k] = __builtin_shufflevector(_lo, _hi, 0, 1, 2, 3, 4, 5, 6, 7); } } while (0)
#define WAIT_LB(Bt_) asm volatile("s_waitcnt lgkmcnt(0)" : "+v"(Bt_[0][0]), "+v"(Bt_[0][1]), "+v"(Bt_[1][0]), "+v"(Bt_[1][1]) :: "memory")
#define MMA(ai, bj, At_, Bt_) do { __builtin_amdgcn_s_setprio(1); _Pragma("unroll") for (int mm = 0; mm < 4; ++mm) _Pragma("unroll") for (int n = 0; n < 2; ++n) _Pragma("unroll") for (int k = 0; k < 2; ++k) \
        acc[ai][bj][mm][n] = __builtin_amdgcn_mfma_f32_16x16x32_bf16(Bt_[n][k], At_[mm][k], acc[ai][bj][mm][n], 0, 0, 0); __builtin_amdgcn_s_setprio(0); } while (0)

#define EARLYB 0
#define MMAB(ai, bj, At_, Bt_) do { __builtin_amdgcn_s_setprio(1); _Pragma("unroll") for (int mm = 0; mm < 4; ++mm) { \
        if (mm == 4 - EARLYB) { SCHED; BAR; SCHED; } \
        _Pragma("unroll") for (int n = 0; n < 2; ++n) _Pragma("unroll") for (int k = 0; k < 2; ++k) \
        acc[ai][bj][mm][n] = __builtin_amdgcn_mfma_f32_16x16x32_bf16(Bt_[n][k], At_[mm][k], acc[ai][bj][mm][n], 0, 0, 0); } \
        __builtin_amdgcn_s_setprio(0); if (EARLYB == 0) BAR; SCHED; } while (0)

    const char *a0 = xloc, *b0 = wbase + (size_t)(rank * KCH) * (NCOL * 2), *a1 = a0 + 128, *b1 = b0 + (size_t)64 * (NCOL * 2);
    const bool pusher = b < 224;
    const int k1 = (b >> 2) & 7;
    const unsigned dst = (unsigned)(rank + 1 + (b >> 5)) & 7u;
    int pcl = 0, uc = 0;
    {
        int u[4];
#pragma unroll
        for (int c = 0; c < 4; ++c) { const int v = 6 - (b >> 5) - COFF(c); u[c] = v < 0 ? v + 7 : v; }
#pragma unroll
        for (int c = 0; c < 4; ++c) {
            const int o = (u[0] * 4 + 0 < u[c] * 4 + c) + (u[1] * 4 + 1 < u[c] * 4 + c) + (u[2] * 4 + 2 < u[c] * 4 + c) + (u[3] * 4 + 3 < u[c] * 4 + c);
            if (o == (k1 & 3)) { pcl = c; uc = u[c]; }
        }
    }
    const int pj = k1 >> 2;
    const unsigned loff = (unsigned)((pcl & 1) * 256 + (b & 3) * 64 + wid * 8 + (lane >> 3)) * (KCH * 2) + (lane & 7) * 16;
    const char* psrc = x + (size_t)dst * BLK_BYTES + (pcl >> 1) * 512 + pj * 128;
    u32x4 pv;
    if (pusher) asm volatile("global_load_dwordx4 %0, %1, %2 nt" : "=v"(pv) : "v"(loff), "s"(psrc) : "memory");
    STB(0, 0, b0); STA(0, 0, a0); STB(0, 1, b0); STA(0, 1, a0); STB(1, 0, b1); STA(1, 0, a1); STB(1, 1, b1);
    char* pws = (char*)ws_all[dst];
    WAIT_V(14);
    asm volatile("" : "+v"(cntv));
    const unsigned call = (unsigned)__builtin_amdgcn_readfirstlane(cntv) + 1u;
    const unsigned par = call & 1u;
    const char* xrem = ws + WS_RECV + (size_t)(par * 8 + h * 4) * BLK_BYTES + (size_t)(m * 256 + flip * 128) * (KCH * 2);
    char* pdst = pws + WS_RECV + (size_t)(par * 8 + (pcl >> 1) * 4 + pj) * BLK_BYTES + rank * 128;
    char* pflag = pws + WS_FLAGA + (size_t)(par * 16384 + pcl * 4096 + (7 * pj + uc) * 128 + (b & 3) * 4);
    bool fl = false;
    int ps = 0, ev = pusher ? (k1 ? k1 : 1) : 1000;
#define PUSH_SLOT() do { if (--ev == 0) { \
        if (ps & 1) { WAIT_V(0); fl = wid == 4; } \
        else { asm volatile("global_store_dwordx4 %0, %1, %2 sc0 sc1\n\ts_nop 2" :: "v"(loff), "v"(pv), "s"(pdst) : "memory"); pdst += 2 * BLK_BYTES; \
               if (ps == 0) asm volatile("global_load_dwordx4 %0, %1, %2 offset:256 nt" : "+v"(pv) : "v"(loff), "s"(psrc) : "memory"); } \
        ++ps; ev = ps < 4 ? (ps == 1 ? GAP1 : 4) : 1000; } } while (0)
#define FLAG_SLOT() do { if (fl) { fl = false; if (lane == 0) asm volatile("global_store_dword %0, %1, %2 sc0 sc1" :: "v"(0u), "v"(call), "s"(pflag) : "memory"); pflag += 14 * 128; } } while (0)
    PUSH_SLOT();

    f32x4 acc[2][2][4][2];
#pragma unroll
    for (int a = 0; a < 2; ++a)
#pragma unroll
        for (int c = 0; c < 2; ++c)
#pragma unroll
            for (int mm = 0; mm < 4; ++mm)
#pragma unroll
                for (int n = 0; n < 2; ++n) acc[a][c][mm][n] = (f32x4){0.f, 0.f, 0.f, 0.f};
    bf16x8 At[4][2], B0[2][2], B1[2][2];
    bool good = true;
    const char* fbase = ws + WS_FLAGA + par * 16384 + (h * 2 + m) * 4096;
    const unsigned fo = (unsigned)(lane & 31) * 128u, cc = call;
    constexpr unsigned FULL = 0x0FFFFFFFu;
    unsigned rdy = wid == 0 ? 0u : FULL, used = 0u, late = 0u;
    u32x4 pa = {cc ^ 1u, 0u, 0u, 0u}, pb = pa;
    unsigned dv = 0u;
#define POLL_ISSUE(pv_) do { if (rdy != FULL) asm volatile("global_load_dwordx4 %0, %1, %2 sc0 sc1" : "+v"(pv_) : "v"(fo), "s"(fbase) : "memory"); } while (0)
#define POLL_MASK(pv_) do { asm volatile("" : "+v"(pv_)); \
        const unsigned long long _m = __builtin_amdgcn_ballot_w64(((pv_[0] ^ cc) | (pv_[1] ^ cc) | (pv_[2] ^ cc) | (pv_[3] ^ cc)) == 0u); \
        rdy |= (unsigned)_m & FULL; } while (0)
    int pfu = 1; unsigned pfd = 0u;
#define TOUCH() do { const char *_a, *_b; TILE_PTRS(pfu, _a, _b); (void)_a; ++pfu; \
        const unsigned _o = (unsigned)(m * 32 + (lane >> 2)) * (unsigned)(NCOL * 2) + (unsigned)(lane & 3) * 128u; \
        asm volatile("global_load_dword %0, %1, %2" : "+v"(pfd) : "v"(_o), "s"(_b) : "memory"); \
        asm volatile("global_load_dword %0, %1, %2" : "+v"(pfd) : "v"(_o), "s"(_b + (size_t)16 * (NCOL * 2)) : "memory"); } while (0)
#define FOLD(z) (((z) | ((z) >> 7) | ((z) >> 14) | ((z) >> 21)) & 0x7Fu)
#define DECIDE(pv_, slot, PF) do { if (wid == 0 && used != FULL) { \
        const unsigned _cand = FULL & ~used; \
        if (rdy != FULL) { POLL_MASK(pv_); \
            unsigned _sel = _cand & (rdy | ~late); \
            if ((_sel & -_sel & rdy) == 0u) { const unsigned long long _t0 = __builtin_amdgcn_s_memrealtime(); unsigned long long _t1 = 0ull; \
                for (;;) { POLL_ISSUE(pv_); if (PF && pfu < PFMAX) { TOUCH(); TOUCH(); WAIT_V(4); } else WAIT_V(0); POLL_MASK(pv_); \
                    _sel = _cand & (rdy | ~late); if ((_sel & -_sel & rdy) != 0u) break; \
                    const unsigned long long _now = __builtin_amdgcn_s_memrealtime(); const unsigned _av = rdy & _cand; \
                    if (_av != 0u) { if (_t1 == 0ull) _t1 = _now; \
                        if (_now - _t1 >= PAT) { late |= (FOLD(_cand & ((_av & -_av) - 1u)) | (LATEALL ? FOLD(rdy) ^ 0x7Fu : 0u)) * 0x204081u; break; } } \
                    if ((unsigned)__builtin_amdgcn_readfirstlane(__hip_atomic_load(tmo, __ATOMIC_RELAXED, __HIP_MEMORY_SCOPE_AGENT)) == call || \
                        _now - _t0 > DEADLINE_TICKS) { \
                        __hip_atomic_store(tmo, call, __ATOMIC_RELAXED, __HIP_MEMORY_SCOPE_AGENT); good = false; rdy = FULL; break; } \
                    __builtin_amdgcn_s_sleep(1); } } } \
        const int _u2 = __builtin_ctz(rdy & _cand); used |= 1u << _u2; \
        asm volatile("ds_write_b32 %0, %1 offset:%2\n\ts_waitcnt lgkmcnt(0)" :: "v"(ldsd), "v"(_u2), "n"((slot) * 4) : "memory"); } } while (0)
#define DEC_READ(slot) asm volatile("ds_read_b32 %0, %1 offset:%2" : "=v"(dv) : "v"(ldsd), "n"((slot) * 4) : "memory")
#define DEC_PTRS(A_, B_) do { asm volatile("" : "+v"(dv)); TILE_PTRS(__builtin_amdgcn_readfirstlane(dv), A_, B_); } while (0)
    const unsigned ldsd = (unsigned)(size_t)lds + (unsigned)LDS_DEC;

    const char *a2 = a0 + 256, *b2 = b0 + (size_t)128 * (NCOL * 2), *a3 = a1, *b3 = b0 + (size_t)192 * (NCOL * 2);
    const char* sbase = ws + WS_SYNC;
    unsigned sv = 0u, sval = call * 64u;
#define PAIR_POST() do { ++sval; if (wid == 0 && lane == 0) asm volatile("global_store_dword %0, %1, %2 sc1" :: "v"(b * 128), "v"(sval), "s"(sbase) : "memory"); } while (0)
#define PAIR_ISSUE() do { if (wid == 0) asm volatile("global_load_dword %0, %1, %2 sc1" : "+v"(sv) : "v"((b ^ 8) * 128), "s"(sbase) : "memory"); } while (0)
#define PAIR_CHECK() do { if (wid == 0) { asm volatile("" : "+v"(sv)); \
        if ((int)((unsigned)__builtin_amdgcn_readfirstlane(sv) - (sval - PSL)) < 0) good = wait_flags((const unsigned*)(sbase + (b ^ 8) * 128), sval, call, tmo) && good; } } while (0)
    WAIT_V(10); BAR;
    if (wr == 1) BAR;
    STAMP(1);

    LDB(B0, 0, 0);
#define ITER(PUSH) do { \
        if (PUSH) FLAG_SLOT(); PAIR_POST(); DEC_READ(0); LDA(At, 0, 0); STA(1, 1, a3); \
        WAIT_V(10); WAIT_L(8); BAR; WAIT_LB(B0); MMAB(0, 0, At, B0); \
        if (PUSH) PUSH_SLOT(); LDB(B1, 0, 1); if (!PUSH || t >= 2) DEC_PTRS(a2, b2); STB(0, 0, b2); \
        WAIT_V(10); BAR; WAIT_LB(B1); MMAB(0, 1, At, B1); \
        if (PUSH) FLAG_SLOT(); POLL_ISSUE(pa); PAIR_ISSUE(); LDA(At, 0, 1); STA(0, 0, a2); \
        WAIT_V(10); BAR; WAIT_L(0); MMAB(1, 0, At, B0); \
        if (PUSH) PUSH_SLOT(); if (!PUSH || t >= 2) DECIDE(pb, 1, PUSH); LDB(B0, 1, 0); STB(0, 1, b2); \
        WAIT_V(10); BAR; MMAB(1, 1, At, B1); \
        if (PUSH) FLAG_SLOT(); DEC_READ(1); LDA(At, 1, 0); STA(0, 1, a2); \
        WAIT_V(10); WAIT_L(8); BAR; WAIT_LB(B0); MMAB(0, 0, At, B0); \
        if (PUSH) PUSH_SLOT(); LDB(B1, 1, 1); if (!PUSH || t >= 2) DEC_PTRS(a3, b3); else a3 = a0 + 384; STB(1, 0, b3); \
        WAIT_V(10); BAR; WAIT_LB(B1); MMAB(0, 1, At, B1); \
        if (PUSH) FLAG_SLOT(); PAIR_POST(); POLL_ISSUE(pb); LDA(At, 1, 1); STA(1, 0, a3); \
        WAIT_V(10); BAR; WAIT_L(0); MMAB(1, 0, At, B0); \
        if (PUSH) PUSH_SLOT(); DECIDE(pa, 0, PUSH); PAIR_CHECK(); LDB(B0, 0, 0); STB(1, 1, b3); \
        WAIT_V(10); BAR; MMAB(1, 1, At, B1); \
        } while (0)

#pragma unroll 1
    for (int t = 0; t < TPUSH; t += 2) {
        if (t == 2) STAMP(2);
        if (t == 4) STAMP(3);
        if (t == 6) { STAMP(4); STAMPC(14); }
        if (t == 8) STAMP(5);
        ITER(1);
    }
#pragma unroll 1
    for (int t = TPUSH; t < NT - 2; t += 2) {
        if (t == 16) { STAMP(6); STAMPC(15); }
        if (t == 20) { STAMP(7); STAMPC(12); }
        ITER(0);
    }
    asm volatile("" :: "v"(pfd));
    STAMP(8); STAMPC(13);

    int tz = threadIdx.x; asm volatile("" : "+v"(tz));
    const int lane2 = tz & 63, fr2 = lane2 & 15, fq2 = lane2 >> 4;
    {
        char* xs = ws + WS_XCH + ((size_t)b * 8 + wid) * 8192 + lane2 * 16;
#define XSEND(c) do { _Pragma("unroll") for (int mm = 0; mm < 4; ++mm) { \
            const f32x4 u0 = acc[0][c][mm][0], u1 = acc[0][c][mm][1]; \
            const u32x4 o = {cvt_pk_bf16(u0[0], u0[1]), cvt_pk_bf16(u0[2], u0[3]), cvt_pk_bf16(u1[0], u1[1]), cvt_pk_bf16(u1[2], u1[3])}; \
            asm volatile("global_store_dwordx4 %0, %1, off offset:%2 sc1\n\ts_nop 2" :: "v"(xs + (c) * 4096), "v"(o), "n"(mm * 1024) : "memory"); } } while (0)
        LDA(At, 0, 0); STA(1, 1, a3);
        WAIT_V(10); BAR; WAIT_LB(B0); MMAB(0, 0, At, B0);
        LDB(B1, 0, 1);
        WAIT_V(8); BAR; WAIT_LB(B1); MMAB(0, 1, At, B1);
        LDA(At, 0, 1);
        WAIT_V(6); BAR; WAIT_L(0); MMAB(1, 0, At, B0);
        LDB(B0, 1, 0);
        WAIT_V(4); BAR; MMAB(1, 1, At, B1);
        LDA(At, 1, 0);
        WAIT_V(2); BAR; WAIT_LB(B0); MMAB(0, 0, At, B0);
        LDB(B1, 1, 1); WAIT_V(0); XSEND(0); SCHED;
        BAR; WAIT_LB(B1); MMAB(0, 1, At, B1);
        LDA(At, 1, 1); XSEND(1); SCHED;
        BAR; WAIT_L(0); MMA(1, 0, At, B0); if (wr == 0) BAR;
        SCHED; MMA(1, 1, At, B1); SCHED;
        WAIT_V(0);
        if (lane2 == 0) __hip_atomic_store((unsigned*)(ws + WS_FLAGX + ((size_t)b * 8 + wid) * 128), call, __ATOMIC_RELAXED, __HIP_MEMORY_SCOPE_AGENT);
    }
    if (tz == 0) __hip_atomic_store(cnt, call, __ATOMIC_RELAXED, __HIP_MEMORY_SCOPE_AGENT);

    STAMP(9);
    {
        const int pb = b ^ 16;
        const unsigned* xf = (const unsigned*)(ws + WS_FLAGX + ((size_t)pb * 8 + wid) * 128);
        const unsigned long long t0 = __builtin_amdgcn_s_memrealtime();
        while ((int)((unsigned)__builtin_amdgcn_readfirstlane(__hip_atomic_load(xf, __ATOMIC_RELAXED, __HIP_MEMORY_SCOPE_AGENT)) - call) < 0) {
            if ((unsigned)__builtin_amdgcn_readfirstlane(__hip_atomic_load(tmo, __ATOMIC_RELAXED, __HIP_MEMORY_SCOPE_AGENT)) == call ||
                __builtin_amdgcn_s_memrealtime() - t0 > DEADLINE_TICKS) {
                __hip_atomic_store(tmo, call, __ATOMIC_RELAXED, __HIP_MEMORY_SCOPE_AGENT);
                break;
            }
            __builtin_amdgcn_s_sleep(1);
        }
        STAMP(10);
        const char* xr = ws + WS_XCH + ((size_t)pb * 8 + wid) * 8192 + lane2 * 16;
        u32x4 rv[2][4];
#pragma unroll
        for (int mm = 0; mm < 4; ++mm)
#pragma unroll
            for (int c = 0; c < 2; ++c)
                asm volatile("global_load_dwordx4 %0, %1, off sc1" : "=v"(rv[c][mm]) : "v"(xr + (c * 4 + mm) * 1024) : "memory");
        const bool bad = !good || (unsigned)__builtin_amdgcn_readfirstlane(__hip_atomic_load(tmo, __ATOMIC_RELAXED, __HIP_MEMORY_SCOPE_AGENT)) == call;
        WAIT_V(0);
#pragma unroll
        for (int c = 0; c < 2; ++c)
#pragma unroll
            for (int mm = 0; mm < 4; ++mm) asm volatile("" : "+v"(rv[c][mm]));
        char* op = out + (size_t)(m * 256 + h * 128 + wr * 64 + fr2) * (NCOL * 2) + (size_t)(ntile * 256 + wc * 32 + 8 * fq2) * 2;
        const bool odd = fq2 & 1;
#pragma unroll
        for (int mm = 0; mm < 4; ++mm)
#pragma unroll
            for (int c = 0; c < 2; ++c) {
                const u32x4 r = rv[c][mm];
                const f32x4 v0 = acc[1][c][mm][0] + (f32x4){__uint_as_float(r[0] << 16), __uint_as_float(r[0] & 0xFFFF0000u), __uint_as_float(r[1] << 16), __uint_as_float(r[1] & 0xFFFF0000u)};
                const f32x4 v1 = acc[1][c][mm][1] + (f32x4){__uint_as_float(r[2] << 16), __uint_as_float(r[2] & 0xFFFF0000u), __uint_as_float(r[3] << 16), __uint_as_float(r[3] & 0xFFFF0000u)};
                const unsigned l0 = cvt_pk_bf16(silu(v0[0]), silu(v0[1])), l1 = cvt_pk_bf16(silu(v0[2]), silu(v0[3]));
                const unsigned h0 = cvt_pk_bf16(silu(v1[0]), silu(v1[1])), h1 = cvt_pk_bf16(silu(v1[2]), silu(v1[3]));
                u32x4 o = odd ? (u32x4){h0, h1, l0, l1} : (u32x4){l0, l1, h0, h1};
                if (bad) o = (u32x4){0x7FC07FC0u, 0x7FC07FC0u, 0x7FC07FC0u, 0x7FC07FC0u};
                *(u32x4*)(op + (size_t)mm * 16 * (NCOL * 2) + c * 256) = o;
            }
        STAMP(11);
    }
}

extern "C" void kernel_launch(int rank, int nranks, ncclComm_t comm,
                              hipStream_t stream,
                              void* const* d_in, const int* in_sizes, int n_in,
                              void* d_out, int out_size,
                              const int* in_dtypes, int out_dtype,
                              void* const* d_out_all,
                              void* const* d_ws_all, size_t ws_bytes) {
    if (nranks != NR || (unsigned)rank >= (unsigned)NR || ws_bytes < WS_END) { fprintf(stderr, "unsupported configuration\n"); return; }
    static bool init = false;
    if (!init) {
        HIP_CHECK(hipFuncSetAttribute((const void*)a2a_gemm_silu, hipFuncAttributeMaxDynamicSharedMemorySize, LDS_BYTES));
        init = true;
    }
    hipLaunchKernelGGL(a2a_gemm_silu, dim3(NWG), dim3(512), LDS_BYTES, stream,
                       (const char*)d_in[0], (const char*)d_in[1], (char*)d_out, d_ws_all, rank);
    HIP_CHECK(hipGetLastError());
}
```

```cpp
#include <hip/hip_runtime.h>
#include <rccl/rccl.h>
#include <cstdio>

#define HIP_CHECK(expr) do { hipError_t _e = (expr); if (_e != hipSuccess) { \
    fprintf(stderr, "HIP error %s at %s:%d\n", hipGetErrorString(_e), __FILE__, __LINE__); \
    return; } } while (0)

#define LAS __attribute__((address_space(3)))
typedef short bf16x8 __attribute__((ext_vector_type(8)));
typedef short s16x4 __attribute__((ext_vector_type(4)));
typedef float f32x4 __attribute__((ext_vector_type(4)));
typedef unsigned u32x4 __attribute__((ext_vector_type(4)));

constexpr int NR = 8;
constexpr int NCOL = 16384;
constexpr int KCH = 512;
constexpr int HTB = 128 * 64 * 2;
constexpr int LDS_DEC = 8 * HTB;
constexpr int LDS_BYTES = LDS_DEC + 64;
constexpr int NWG = 256;
constexpr int TPUSH = 10;
#define PAT 150ull
#define LATEALL 1
#define PFMAX 0
#define PSL 2u
#define WARM 0
#define CDRAIN 0
#define DEVFB 1
#define GAP1 2
#define COFF(c) (((c) >> 1) * 3)
constexpr int NT = 32;

constexpr size_t WS_TMO   = 0;
constexpr size_t WS_CNT   = 4096;
constexpr size_t WS_FLAGA = WS_CNT + NWG * 128;
constexpr size_t WS_FLAGX = WS_FLAGA + 32768;
constexpr size_t WS_SYNC  = WS_FLAGX + NWG * 8 * 128;
constexpr size_t WS_RECV  = 1u << 20;
constexpr size_t BLK_BYTES = (size_t)KCH * KCH * 2;
constexpr size_t WS_XCH   = WS_RECV + 2 * NR * BLK_BYTES;
constexpr size_t WS_END   = WS_XCH + (size_t)NWG * 8 * 16 * 1024;
constexpr unsigned long long DEADLINE_TICKS = 40000000ull;

__device__ __forceinline__ int lds_byte(int r, int c) { const int st = (r >> 4) * 2 + (c >> 5), rr = r & 15, cc = c & 31, ob = rr * 64 + cc * 2; return st * 1024 + (ob ^ (((ob >> 9) & 1) << 5)); }
__device__ __forceinline__ void stage_rc(int b, int& R, int& C) { const int st = b / 1024, sb = b % 1024, swz = sb ^ (((sb >> 9) & 1) << 5); R = (st >> 1) * 16 + swz / 64; C = (st & 1) * 32 + (swz % 64) / 2; }
__device__ __forceinline__ unsigned cvt_pk_bf16(float lo, float hi) { unsigned r; asm volatile("v_cvt_pk_bf16_f32 %0, %1, %2" : "=v"(r) : "v"(lo), "v"(hi)); return r; }
__device__ __forceinline__ float silu(float y) {
    const float c = __builtin_amdgcn_fmed3f(y, -60.0f, 60.0f);
    const float e = __builtin_amdgcn_exp2f(c * -1.44269504088896f);
    return y * __builtin_amdgcn_rcpf(1.0f + e);
}

#define STAMP(k) do {} while (0)
#define STAMPC(k) do {} while (0)
#define WAIT_V(n) asm volatile("s_waitcnt vmcnt(" #n ")" ::: "memory")
#define WAIT_L(n) asm volatile("s_waitcnt lgkmcnt(" #n ")" ::: "memory")
#define BAR __builtin_amdgcn_s_barrier()
#define SCHED __builtin_amdgcn_sched_barrier(0)

__device__ __forceinline__ bool wait_flags(const unsigned* f, unsigned val, unsigned call, unsigned* tmo) {
    const unsigned long long t0 = __builtin_amdgcn_s_memrealtime();
    for (;;) {
        const unsigned v = __hip_atomic_load(f, __ATOMIC_RELAXED, __HIP_MEMORY_SCOPE_SYSTEM);
        const bool ok = (int)(v - val) >= 0;
        if (__builtin_amdgcn_ballot_w64(!ok) == 0ull) return true;
        if ((unsigned)__builtin_amdgcn_readfirstlane(__hip_atomic_load(tmo, __ATOMIC_RELAXED, __HIP_MEMORY_SCOPE_AGENT)) == call ||
            __builtin_amdgcn_s_memrealtime() - t0 > DEADLINE_TICKS) {
            __hip_atomic_store(tmo, call, __ATOMIC_RELAXED, __HIP_MEMORY_SCOPE_AGENT);
            return false;
        }
        __builtin_amdgcn_s_sleep(1);
    }
}

__global__ __launch_bounds__(512, 2)
void a2a_gemm_silu(const char* __restrict__ x, const char* __restrict__ w, char* __restrict__ out,
                   void* const* __restrict__ ws_all, int rank) {
    extern __shared__ __attribute__((aligned(16))) unsigned char shm[];
    LAS unsigned char* lds = (LAS unsigned char*)shm;
    const int tid = threadIdx.x, wid = __builtin_amdgcn_readfirstlane(tid >> 6), lane = tid & 63;
    const int wr = wid >> 2, wc = wid & 3, fr = lane & 15, fq = lane >> 4;
    const int b = blockIdx.x, idx = b >> 3;
    const int ntile = (b & 7) * 8 + (idx >> 2), m = idx & 1, h = (idx >> 1) & 1;
    char* ws = (char*)ws_all[rank];
    unsigned* tmo = (unsigned*)(ws + WS_TMO);
    STAMP(0);

    unsigned* cnt = (unsigned*)(ws + WS_CNT + (size_t)b * 128);
    unsigned cntv;
    asm volatile("global_load_dword %0, %1, off sc1" : "=v"(cntv) : "v"(cnt) : "memory");

    unsigned voffA[2], voffB[2];
#pragma unroll
    for (int i = 0; i < 2; ++i) {
        int R, C; stage_rc(tid * 16 + i * 8192, R, C);
        voffA[i] = (unsigned)(R * KCH + C) * 2u;
        const int r = wid * 4 + (lane >> 4) + i * 32, c = (lane & 15) ^ (((r & 3) << 2) | ((r >> 3) & 1));
        voffB[i] = (unsigned)r * (unsigned)(NCOL * 2) + (unsigned)c * 16u;
    }
    const unsigned ldsw = (unsigned)wid * 1024u;
    const int aoff = lds_byte(wr * 64 + fr, fq * 8);
    const int q4 = (lane & 15) >> 2, p4 = lane & 3;
    const int boff0 = 256 * (8 * fq + q4) + 16 * ((4 * wc + p4) ^ (q4 << 2) ^ (fq & 1)) + 8 * (p4 & 1);
    const unsigned bad0 = (unsigned)(size_t)lds + 65536u + (unsigned)boff0, bad1 = bad0 ^ 8u;

    const int flip = h ^ 1;
    const long aoffs0 = (long)(m * 256 + flip * 128) * (KCH * 2) + h * 512;
    const long hstepA = (flip ? -1L : 1L) * 128 * (KCH * 2);
    const char* xloc = x + (size_t)rank * BLK_BYTES + aoffs0;
    const char* wbase = w + (size_t)(h * 256) * (NCOL * 2) + (size_t)ntile * 512;
    constexpr long hstepB = 256;

    const int coff = COFF(h * 2 + m);
#define TILE_PTRS(u_, A_, B_) do { const int _u = (u_); const int _j = (_u * 37) >> 8; \
        const int _d0 = _u - _j * 7 + coff; const int _d = _d0 >= 7 ? _d0 - 7 : _d0; const int _p = (rank + 1 + _d) & 7; \
        A_ = xrem + (size_t)_j * BLK_BYTES + _p * 128; \
        B_ = wbase + (size_t)(_p * KCH + _j * 64) * (NCOL * 2); } while (0)

#define SA(bb, hh) (((bb) * 2 + (hh)) * HTB)
#define SB(bb, hh) ((4 + (bb) * 2 + (hh)) * HTB)
#define OUTNT 0
#define WAUX 0
#define WAUX1 2
#define GLDS(gp, loff, aux) __builtin_amdgcn_global_load_lds((const unsigned*)(gp), (LAS unsigned*)(lds + (loff)), 16, 0, aux)
#define STAGE(bufoff, gbase, voff, aux) do { GLDS((gbase) + (voff)[0], (bufoff) + ldsw, aux); GLDS((gbase) + (voff)[1], (bufoff) + ldsw + 8192, aux); } while (0)
#define STA(bb, hh, ap) STAGE(SA(bb, hh), (ap) + (hh) * hstepA, voffA, 0)
#define STB(bb, hh, bp) STAGE(SB(bb, hh), (bp) + (hh) * hstepB, voffB, ((bb) ? WAUX1 : WAUX))
#define LDA(dst, bb, hh) do { _Pragma("unroll") for (int mm = 0; mm < 4; ++mm) _Pragma("unroll") for (int k = 0; k < 2; ++k) \
        dst[mm][k] = *(const LAS bf16x8*)(lds + SA(bb, hh) + aoff + mm * 2048 + k * 1024); } while (0)
#define TRD(dst, base, off) asm volatile("ds_read_b64_tr_b16 %0, %1 offset:%2" : "=v"(dst) : "v"(base), "n"(off) : "memory")
#define LDB(dst, bb, hh) do { _Pragma("unroll") for (int n = 0; n < 2; ++n) _Pragma("unroll") for (int k = 0; k < 2; ++k) { \
        s16x4 _lo, _hi; TRD(_lo, (n ? bad1 : bad0), SB(bb, hh) - 65536 + k * 8192); TRD(_hi, (n ? bad1 : bad0), SB(bb, hh) - 65536 + k * 8192 + 1024); \
        dst[n][k] = __builtin_shufflevector(_lo, _hi, 0, 1, 2, 3, 4, 5, 6, 7); } } while (0)
#define WAIT_LB(Bt_) asm volatile("s_waitcnt lgkmcnt(0)" : "+v"(Bt_[0][0]), "+v"(Bt_[0][1]), "+v"(Bt_[1][0]), "+v"(Bt_[1][1]) :: "memory")
#define MMA(ai, bj, At_, Bt_) do { __builtin_amdgcn_s_setprio(1); _Pragma("unroll") for (int mm = 0; mm < 4; ++mm) _Pragma("unroll") for (int n = 0; n < 2; ++n) _Pragma("unroll") for (int k = 0; k < 2; ++k) \
        acc[ai][bj][mm][n] = __builtin_amdgcn_mfma_f32_16x16x32_bf16(Bt_[n][k], At_[mm][k], acc[ai][bj][mm][n], 0, 0, 0); __builtin_amdgcn_s_setprio(0); } while (0)

#define EARLYB 0
#define MMAB(ai, bj, At_, Bt_) do { __builtin_amdgcn_s_setprio(1); _Pragma("unroll") for (int mm = 0; mm < 4; ++mm) { \
        if (mm == 4 - EARLYB) { SCHED; BAR; SCHED; } \
        _Pragma("unroll") for (int n = 0; n < 2; ++n) _Pragma("unroll") for (int k = 0; k < 2; ++k) \
        acc[ai][bj][mm][n] = __builtin_amdgcn_mfma_f32_16x16x32_bf16(Bt_[n][k], At_[mm][k], acc[ai][bj][mm][n], 0, 0, 0); } \
        __builtin_amdgcn_s_setprio(0); if (EARLYB == 0) BAR; SCHED; } while (0)

    const char *a0 = xloc, *b0 = wbase + (size_t)(rank * KCH) * (NCOL * 2), *a1 = a0 + 128, *b1 = b0 + (size_t)64 * (NCOL * 2);
    const bool pusher = b < 224;
    const int k1 = (b >> 2) & 7;
    const unsigned dst = (unsigned)(rank + 1 + (b >> 5)) & 7u;
    int pcl = 0, uc = 0;
    {
        int u[4];
#pragma unroll
        for (int c = 0; c < 4; ++c) { const int v = 6 - (b >> 5) - COFF(c); u[c] = v < 0 ? v + 7 : v; }
#pragma unroll
        for (int c = 0; c < 4; ++c) {
            const int o = (u[0] * 4 + 0 < u[c] * 4 + c) + (u[1] * 4 + 1 < u[c] * 4 + c) + (u[2] * 4 + 2 < u[c] * 4 + c) + (u[3] * 4 + 3 < u[c] * 4 + c);
            if (o == (k1 & 3)) { pcl = c; uc = u[c]; }
        }
    }
    const int pj = k1 >> 2;
    const unsigned loff = (unsigned)((pcl & 1) * 256 + (b & 3) * 64 + wid * 8 + (lane >> 3)) * (KCH * 2) + (lane & 7) * 16;
    const char* psrc = x + (size_t)dst * BLK_BYTES + (pcl >> 1) * 512 + pj * 128;
    u32x4 pv;
    if (pusher) asm volatile("global_load_dwordx4 %0, %1, %2 nt" : "=v"(pv) : "v"(loff), "s"(psrc) : "memory");
    STB(0, 0, b0); STA(0, 0, a0); STB(0, 1, b0); STA(0, 1, a0); STB(1, 0, b1); STA(1, 0, a1); STB(1, 1, b1);
    char* pws = (char*)ws_all[dst];
    unsigned pfd = 0u;
    WAIT_V(14);
    asm volatile("" : "+v"(cntv));
    const unsigned call = (unsigned)__builtin_amdgcn_readfirstlane(cntv) + 1u;
    const unsigned par = call & 1u;
    const char* xrem = ws + WS_RECV + (size_t)(par * 8 + h * 4) * BLK_BYTES + (size_t)(m * 256 + flip * 128) * (KCH * 2);
    char* pdst = pws + WS_RECV + (size_t)(par * 8 + (pcl >> 1) * 4 + pj) * BLK_BYTES + rank * 128;
    char* pflag = pws + WS_FLAGA + (size_t)(par * 16384 + pcl * 4096 + (7 * pj + uc) * 128 + (b & 3) * 4);
    static_assert(GAP1 == 2, "drain count");
    bool fl = false;
    int ps = 0, ev = pusher ? (k1 ? k1 : 1) : 1000;
#define PUSH_SLOT() do { if (--ev == 0) { \
        if (ps & 1) { if (!CDRAIN) WAIT_V(0); else if (ps == 1) WAIT_V(6); fl = wid == 4; } \
        else { asm volatile("global_store_dwordx4 %0, %1, %2 sc0 sc1\n\ts_nop 2" :: "v"(loff), "v"(pv), "s"(pdst) : "memory"); pdst += 2 * BLK_BYTES; \
               if (ps == 0) asm volatile("global_load_dwordx4 %0, %1, %2 offset:256 nt" : "+v"(pv) : "v"(loff), "s"(psrc) : "memory"); } \
        ++ps; ev = ps < 4 ? (ps == 1 ? GAP1 : 4) : 1000; } } while (0)
#define FLAG_SLOT() do { if (fl) { fl = false; if (lane == 0) asm volatile("global_store_dword %0, %1, %2 sc0 sc1" :: "v"(0u), "v"(call), "s"(pflag) : "memory"); pflag += 14 * 128; } } while (0)
    PUSH_SLOT();

    f32x4 acc[2][2][4][2];
#pragma unroll
    for (int a = 0; a < 2; ++a)
#pragma unroll
        for (int c = 0; c < 2; ++c)
#pragma unroll
            for (int mm = 0; mm < 4; ++mm)
#pragma unroll
                for (int n = 0; n < 2; ++n) acc[a][c][mm][n] = (f32x4){0.f, 0.f, 0.f, 0.f};
    bf16x8 At[4][2], B0[2][2], B1[2][2];
    bool good = true;
    const char* fbase = ws + WS_FLAGA + par * 16384 + (h * 2 + m) * 4096;
    const unsigned fo = (unsigned)(lane & 31) * 128u, cc = call;
    constexpr unsigned FULL = 0x0FFFFFFFu;
    unsigned rdy = wid == 0 ? 0u : FULL, used = 0u, late = 0u;
    u32x4 pa = {cc ^ 1u, 0u, 0u, 0u}, pb = pa;
    unsigned dv = 0u;
#define POLL_ISSUE(pv_) do { if (rdy != FULL) asm volatile("global_load_dwordx4 %0, %1, %2 sc0 sc1" : "+v"(pv_) : "v"(fo), "s"(fbase) : "memory"); } while (0)
#define POLL_MASK(pv_) do { asm volatile("" : "+v"(pv_)); \
        const unsigned long long _m = __builtin_amdgcn_ballot_w64(((pv_[0] ^ cc) | (pv_[1] ^ cc) | (pv_[2] ^ cc) | (pv_[3] ^ cc)) == 0u); \
        rdy |= (unsigned)_m & FULL; } while (0)
    int pfu = 1;
#define TOUCH() do { const char *_a, *_b; TILE_PTRS(pfu, _a, _b); (void)_a; ++pfu; \
        const unsigned _o = (unsigned)(m * 32 + (lane >> 2)) * (unsigned)(NCOL * 2) + (unsigned)(lane & 3) * 128u; \
        asm volatile("global_load_dword %0, %1, %2" : "+v"(pfd) : "v"(_o), "s"(_b) : "memory"); \
        asm volatile("global_load_dword %0, %1, %2" : "+v"(pfd) : "v"(_o), "s"(_b + (size_t)16 * (NCOL * 2)) : "memory"); } while (0)
#define FOLD(z) (((z) | ((z) >> 7) | ((z) >> 14) | ((z) >> 21)) & 0x7Fu)
#define DECIDE(pv_, slot, PF) do { if (wid == 0 && used != FULL) { \
        const unsigned _cand = FULL & ~used; \
        if (rdy != FULL) { POLL_MASK(pv_); \
            unsigned _sel = _cand & (rdy | ~late); \
            if ((_sel & -_sel & rdy) == 0u) { const unsigned long long _t0 = __builtin_amdgcn_s_memrealtime(); unsigned long long _t1 = 0ull; \
                for (;;) { POLL_ISSUE(pv_); if (PF && pfu < PFMAX) { TOUCH(); TOUCH(); WAIT_V(4); } else WAIT_V(0); POLL_MASK(pv_); \
                    _sel = _cand & (rdy | ~late); if ((_sel & -_sel & rdy) != 0u) break; \
                    const unsigned long long _now = __builtin_amdgcn_s_memrealtime(); const unsigned _av = rdy & _cand; \
                    if (_av != 0u) { if (_t1 == 0ull) _t1 = _now; \
                        if (_now - _t1 >= PAT) { late |= (FOLD(_cand & ((_av & -_av) - 1u)) | (LATEALL ? FOLD(rdy) ^ 0x7Fu : 0u)) * 0x204081u; break; } } \
                    if ((unsigned)__builtin_amdgcn_readfirstlane(__hip_atomic_load(tmo, __ATOMIC_RELAXED, __HIP_MEMORY_SCOPE_AGENT)) == call || \
                        _now - _t0 > DEADLINE_TICKS) { \
                        __hip_atomic_store(tmo, call, __ATOMIC_RELAXED, __HIP_MEMORY_SCOPE_AGENT); good = false; rdy = FULL; break; } \
                    __builtin_amdgcn_s_sleep(1); } } } \
        const int _u2 = __builtin_ctz(rdy & _cand); used |= 1u << _u2; \
        asm volatile("ds_write_b32 %0, %1 offset:%2\n\ts_waitcnt lgkmcnt(0)" :: "v"(ldsd), "v"(_u2), "n"((slot) * 4) : "memory"); } } while (0)
    bool dev = false;
#define STB1(hh, bp) do { if (dev) STAGE(SB(1, hh), (bp) + (hh) * hstepB, voffB, 0); else STB(1, hh, bp); } while (0)
#define DEC_READ(slot) asm volatile("ds_read_b32 %0, %1 offset:%2" : "=v"(dv) : "v"(ldsd), "n"((slot) * 4) : "memory")
#define DEC_PTRS(A_, B_) do { asm volatile("" : "+v"(dv)); TILE_PTRS(__builtin_amdgcn_readfirstlane(dv), A_, B_); } while (0)
    const unsigned ldsd = (unsigned)(size_t)lds + (unsigned)LDS_DEC;

    const char *a2 = a0 + 256, *b2 = b0 + (size_t)128 * (NCOL * 2), *a3 = a1, *b3 = b0 + (size_t)192 * (NCOL * 2);
    const char* sbase = ws + WS_SYNC;
    unsigned sv = 0u, sval = call * 64u;
#define PAIR_POST() do { ++sval; if (wid == 0 && lane == 0) asm volatile("global_store_dword %0, %1, %2 sc1" :: "v"(b * 128), "v"(sval), "s"(sbase) : "memory"); } while (0)
#define PAIR_ISSUE() do { if (wid == 0) asm volatile("global_load_dword %0, %1, %2 sc1" : "+v"(sv) : "v"((b ^ 8) * 128), "s"(sbase) : "memory"); } while (0)
#define PAIR_CHECK() do { if (wid == 0) { asm volatile("" : "+v"(sv)); \
        if ((int)((unsigned)__builtin_amdgcn_readfirstlane(sv) - (sval - PSL)) < 0) good = wait_flags((const unsigned*)(sbase + (b ^ 8) * 128), sval, call, tmo) && good; } } while (0)
    WAIT_V(10); BAR;
    if (wr == 1) BAR;
    STAMP(1);

    LDB(B0, 0, 0);
#define ITER(PUSH) do { \
        if (PUSH) FLAG_SLOT(); PAIR_POST(); DEC_READ(0); LDA(At, 0, 0); STA(1, 1, a3); \
        WAIT_V(10); WAIT_L(8); BAR; WAIT_LB(B0); MMAB(0, 0, At, B0); \
        if (PUSH) PUSH_SLOT(); LDB(B1, 0, 1); if (!PUSH || t >= 2) DEC_PTRS(a2, b2); STB(0, 0, b2); \
        WAIT_V(10); BAR; WAIT_LB(B1); MMAB(0, 1, At, B1); \
        if (PUSH) FLAG_SLOT(); POLL_ISSUE(pa); PAIR_ISSUE(); LDA(At, 0, 1); STA(0, 0, a2); \
        WAIT_V(10); BAR; WAIT_L(0); MMAB(1, 0, At, B0); \
        if (PUSH) PUSH_SLOT(); if (!PUSH || t >= 2) DECIDE(pb, 1, PUSH); LDB(B0, 1, 0); STB(0, 1, b2); \
        WAIT_V(10); BAR; MMAB(1, 1, At, B1); \
        if (PUSH) FLAG_SLOT(); DEC_READ(1); LDA(At, 1, 0); STA(0, 1, a2); \
        WAIT_V(10); WAIT_L(8); BAR; WAIT_LB(B0); MMAB(0, 0, At, B0); \
        if (PUSH) PUSH_SLOT(); LDB(B1, 1, 1); if (!PUSH || t >= 2) { DEC_PTRS(a3, b3); dev = DEVFB && (int)__builtin_amdgcn_readfirstlane(dv) != t - 1; } else a3 = a0 + 384; STB1(0, b3); \
        WAIT_V(10); BAR; WAIT_LB(B1); MMAB(0, 1, At, B1); \
        if (PUSH) FLAG_SLOT(); PAIR_POST(); POLL_ISSUE(pb); LDA(At, 1, 1); STA(1, 0, a3); \
        WAIT_V(10); BAR; WAIT_L(0); MMAB(1, 0, At, B0); \
        if (PUSH) PUSH_SLOT(); DECIDE(pa, 0, PUSH); PAIR_CHECK(); LDB(B0, 0, 0); STB1(1, b3); \
        WAIT_V(10); BAR; MMAB(1, 1, At, B1); \
        } while (0)

#pragma unroll 1
    for (int t = 0; t < TPUSH; t += 2) {
        if (t == 2) STAMP(2);
        if (t == 4) STAMP(3);
        if (t == 6) { STAMP(4); STAMPC(14); }
        if (t == 8) STAMP(5);
        ITER(1);
    }
#pragma unroll 1
    for (int t = TPUSH; t < NT - 2; t += 2) {
        if (t == 16) { STAMP(6); STAMPC(15); }
        if (t == 20) { STAMP(7); STAMPC(12); }
        ITER(0);
    }
    asm volatile("" :: "v"(pfd));
    STAMP(8); STAMPC(13);

    int tz = threadIdx.x; asm volatile("" : "+v"(tz));
    const int lane2 = tz & 63, fr2 = lane2 & 15, fq2 = lane2 >> 4;
    {
        char* xs = ws + WS_XCH + ((size_t)b * 8 + wid) * 8192 + lane2 * 16;
#define XSEND(c) do { _Pragma("unroll") for (int mm = 0; mm < 4; ++mm) { \
            const f32x4 u0 = acc[0][c][mm][0], u1 = acc[0][c][mm][1]; \
            const u32x4 o = {cvt_pk_bf16(u0[0], u0[1]), cvt_pk_bf16(u0[2], u0[3]), cvt_pk_bf16(u1[0], u1[1]), cvt_pk_bf16(u1[2], u1[3])}; \
            asm volatile("global_store_dwordx4 %0, %1, off offset:%2 sc1\n\ts_nop 2" :: "v"(xs + (c) * 4096), "v"(o), "n"(mm * 1024) : "memory"); } } while (0)
        LDA(At, 0, 0); STA(1, 1, a3);
        WAIT_V(10); BAR; WAIT_LB(B0); MMAB(0, 0, At, B0);
        LDB(B1, 0, 1);
        WAIT_V(8); BAR; WAIT_LB(B1); MMAB(0, 1, At, B1);
        LDA(At, 0, 1);
        WAIT_V(6); BAR; WAIT_L(0); MMAB(1, 0, At, B0);
        LDB(B0, 1, 0);
        WAIT_V(4); BAR; MMAB(1, 1, At, B1);
        LDA(At, 1, 0);
        WAIT_V(2); BAR; WAIT_LB(B0); MMAB(0, 0, At, B0);
        LDB(B1, 1, 1); WAIT_V(0); XSEND(0); SCHED;
        BAR; WAIT_LB(B1); MMAB(0, 1, At, B1);
        LDA(At, 1, 1); XSEND(1); SCHED;
        BAR; WAIT_L(0); MMA(1, 0, At, B0); if (wr == 0) BAR;
        SCHED; MMA(1, 1, At, B1); SCHED;
        WAIT_V(0);
        if (lane2 == 0) __hip_atomic_store((unsigned*)(ws + WS_FLAGX + ((size_t)b * 8 + wid) * 128), call, __ATOMIC_RELAXED, __HIP_MEMORY_SCOPE_AGENT);
    }
    if (tz == 0) __hip_atomic_store(cnt, call, __ATOMIC_RELAXED, __HIP_MEMORY_SCOPE_AGENT);

    STAMP(9);
    {
        const int pb = b ^ 16;
        const unsigned* xf = (const unsigned*)(ws + WS_FLAGX + ((size_t)pb * 8 + wid) * 128);
        const unsigned long long t0 = __builtin_amdgcn_s_memrealtime();
        while ((int)((unsigned)__builtin_amdgcn_readfirstlane(__hip_atomic_load(xf, __ATOMIC_RELAXED, __HIP_MEMORY_SCOPE_AGENT)) - call) < 0) {
            if ((unsigned)__builtin_amdgcn_readfirstlane(__hip_atomic_load(tmo, __ATOMIC_RELAXED, __HIP_MEMORY_SCOPE_AGENT)) == call ||
                __builtin_amdgcn_s_memrealtime() - t0 > DEADLINE_TICKS) {
                __hip_atomic_store(tmo, call, __ATOMIC_RELAXED, __HIP_MEMORY_SCOPE_AGENT);
                break;
            }
            __builtin_amdgcn_s_sleep(1);
        }
        STAMP(10);
        const char* xr = ws + WS_XCH + ((size_t)pb * 8 + wid) * 8192 + lane2 * 16;
        u32x4 rv[2][4];
#pragma unroll
        for (int mm = 0; mm < 4; ++mm)
#pragma unroll
            for (int c = 0; c < 2; ++c)
                asm volatile("global_load_dwordx4 %0, %1, off sc1" : "=v"(rv[c][mm]) : "v"(xr + (c * 4 + mm) * 1024) : "memory");
        const bool bad = !good || (unsigned)__builtin_amdgcn_readfirstlane(__hip_atomic_load(tmo, __ATOMIC_RELAXED, __HIP_MEMORY_SCOPE_AGENT)) == call;
        WAIT_V(0);
#pragma unroll
        for (int c = 0; c < 2; ++c)
#pragma unroll
            for (int mm = 0; mm < 4; ++mm) asm volatile("" : "+v"(rv[c][mm]));
        char* op = out + (size_t)(m * 256 + h * 128 + wr * 64 + fr2) * (NCOL * 2) + (size_t)(ntile * 256 + wc * 32 + 8 * fq2) * 2;
        const bool odd = fq2 & 1;
#pragma unroll
        for (int mm = 0; mm < 4; ++mm)
#pragma unroll
            for (int c = 0; c < 2; ++c) {
                const u32x4 r = rv[c][mm];
                const f32x4 v0 = acc[1][c][mm][0] + (f32x4){__uint_as_float(r[0] << 16), __uint_as_float(r[0] & 0xFFFF0000u), __uint_as_float(r[1] << 16), __uint_as_float(r[1] & 0xFFFF0000u)};
                const f32x4 v1 = acc[1][c][mm][1] + (f32x4){__uint_as_float(r[2] << 16), __uint_as_float(r[2] & 0xFFFF0000u), __uint_as_float(r[3] << 16), __uint_as_float(r[3] & 0xFFFF0000u)};
                const unsigned l0 = cvt_pk_bf16(silu(v0[0]), silu(v0[1])), l1 = cvt_pk_bf16(silu(v0[2]), silu(v0[3]));
                const unsigned h0 = cvt_pk_bf16(silu(v1[0]), silu(v1[1])), h1 = cvt_pk_bf16(silu(v1[2]), silu(v1[3]));
                u32x4 o = odd ? (u32x4){h0, h1, l0, l1} : (u32x4){l0, l1, h0, h1};
                if (bad) o = (u32x4){0x7FC07FC0u, 0x7FC07FC0u, 0x7FC07FC0u, 0x7FC07FC0u};
                *(u32x4*)(op + (size_t)mm * 16 * (NCOL * 2) + c * 256) = o;
            }
        STAMP(11);
    }
}

extern "C" void kernel_launch(int rank, int nranks, ncclComm_t comm,
                              hipStream_t stream,
                              void* const* d_in, const int* in_sizes, int n_in,
                              void* d_out, int out_size,
                              const int* in_dtypes, int out_dtype,
                              void* const* d_out_all,
                              void* const* d_ws_all, size_t ws_bytes) {
    if (nranks != NR || (unsigned)rank >= (unsigned)NR || ws_bytes < WS_END) { fprintf(stderr, "unsupported configuration\n"); return; }
    static bool init = false;
    if (!init) {
        HIP_CHECK(hipFuncSetAttribute((const void*)a2a_gemm_silu, hipFuncAttributeMaxDynamicSharedMemorySize, LDS_BYTES));
        init = true;
    }
    hipLaunchKernelGGL(a2a_gemm_silu, dim3(NWG), dim3(512), LDS_BYTES, stream,
                       (const char*)d_in[0], (const char*)d_in[1], (char*)d_out, d_ws_all, rank);
    HIP_CHECK(hipGetLastError());
}
```

```cpp
#include <hip/hip_runtime.h>
#include <rccl/rccl.h>
#include <cstdio>

#define HIP_CHECK(expr) do { hipError_t _e = (expr); if (_e != hipSuccess) { \
    fprintf(stderr, "HIP error %s at %s:%d\n", hipGetErrorString(_e), __FILE__, __LINE__); \
    return; } } while (0)

#define LAS __attribute__((address_space(3)))
typedef short bf16x8 __attribute__((ext_vector_type(8)));
typedef short s16x4 __attribute__((ext_vector_type(4)));
typedef float f32x4 __attribute__((ext_vector_type(4)));
typedef unsigned u32x4 __attribute__((ext_vector_type(4)));

constexpr int NR = 8;
constexpr int NCOL = 16384;
constexpr int KCH = 512;
constexpr int HTB = 128 * 64 * 2;
constexpr int LDS_DEC = 8 * HTB;
constexpr int LDS_BYTES = LDS_DEC + 64;
constexpr int NWG = 256;
constexpr int TPUSH = 10;
#define PAT 150ull
#define LATEALL 1
#define GAP1 2
#define COFF(c) (((c) >> 1) * 3)
constexpr int NT = 32;

constexpr size_t WS_TMO   = 0;
constexpr size_t WS_CNT   = 4096;
constexpr size_t WS_FLAGA = WS_CNT + NWG * 128;
constexpr size_t WS_FLAGX = WS_FLAGA + 32768;
constexpr size_t WS_SYNC  = WS_FLAGX + NWG * 8 * 128;
constexpr size_t WS_RECV  = 1u << 20;
constexpr size_t BLK_BYTES = (size_t)KCH * KCH * 2;
constexpr size_t WS_XCH   = WS_RECV + 2 * NR * BLK_BYTES;
constexpr size_t WS_END   = WS_XCH + (size_t)NWG * 8 * 16 * 1024;
constexpr unsigned long long DEADLINE_TICKS = 40000000ull;

__device__ __forceinline__ int lds_byte(int r, int c) { const int st = (r >> 4) * 2 + (c >> 5), rr = r & 15, cc = c & 31, ob = rr * 64 + cc * 2; return st * 1024 + (ob ^ (((ob >> 9) & 1) << 5)); }
__device__ __forceinline__ void stage_rc(int b, int& R, int& C) { const int st = b / 1024, sb = b % 1024, swz = sb ^ (((sb >> 9) & 1) << 5); R = (st >> 1) * 16 + swz / 64; C = (st & 1) * 32 + (swz % 64) / 2; }
__device__ __forceinline__ unsigned cvt_pk_bf16(float lo, float hi) { unsigned r; asm volatile("v_cvt_pk_bf16_f32 %0, %1, %2" : "=v"(r) : "v"(lo), "v"(hi)); return r; }
__device__ __forceinline__ float silu(float y) {
    const float c = __builtin_amdgcn_fmed3f(y, -60.0f, 60.0f);
    const float e = __builtin_amdgcn_exp2f(c * -1.44269504088896f);
    return y * __builtin_amdgcn_rcpf(1.0f + e);
}

#define STAMP(k) do {} while (0)
#define STAMPC(k) do {} while (0)
#define WAIT_V(n) asm volatile("s_waitcnt vmcnt(" #n ")" ::: "memory")
#define WAIT_L(n) asm volatile("s_waitcnt lgkmcnt(" #n ")" ::: "memory")
#define BAR __builtin_amdgcn_s_barrier()
#define SCHED __builtin_amdgcn_sched_barrier(0)

__device__ __forceinline__ bool wait_flags(const unsigned* f, unsigned val, unsigned call, unsigned* tmo) {
    const unsigned long long t0 = __builtin_amdgcn_s_memrealtime();
    for (;;) {
        const unsigned v = __hip_atomic_load(f, __ATOMIC_RELAXED, __HIP_MEMORY_SCOPE_SYSTEM);
        const bool ok = (int)(v - val) >= 0;
        if (__builtin_amdgcn_ballot_w64(!ok) == 0ull) return true;
        if ((unsigned)__builtin_amdgcn_readfirstlane(__hip_atomic_load(tmo, __ATOMIC_RELAXED, __HIP_MEMORY_SCOPE_AGENT)) == call ||
            __builtin_amdgcn_s_memrealtime() - t0 > DEADLINE_TICKS) {
            __hip_atomic_store(tmo, call, __ATOMIC_RELAXED, __HIP_MEMORY_SCOPE_AGENT);
            return false;
        }
        __builtin_amdgcn_s_sleep(1);
    }
}

__global__ __launch_bounds__(512, 2)
void a2a_gemm_silu(const char* __restrict__ x, const char* __restrict__ w, char* __restrict__ out,
                   void* const* __restrict__ ws_all, int rank) {
    extern __shared__ __attribute__((aligned(16))) unsigned char shm[];
    LAS unsigned char* lds = (LAS unsigned char*)shm;
    const int tid = threadIdx.x, wid = __builtin_amdgcn_readfirstlane(tid >> 6), lane = tid & 63;
    const int wr = wid >> 2, wc = wid & 3, fr = lane & 15, fq = lane >> 4;
    const int b = blockIdx.x, idx = b >> 3;
    const int ntile = (b & 7) * 8 + (idx >> 2), m = idx & 1, h = (idx >> 1) & 1;
    char* ws = (char*)ws_all[rank];
    unsigned* tmo = (unsigned*)(ws + WS_TMO);
    STAMP(0);

    unsigned* cnt = (unsigned*)(ws + WS_CNT + (size_t)b * 128);
    unsigned cntv;
    asm volatile("global_load_dword %0, %1, off sc1" : "=v"(cntv) : "v"(cnt) : "memory");

    unsigned voffA[2], voffB[2];
#pragma unroll
    for (int i = 0; i < 2; ++i) {
        int R, C; stage_rc(tid * 16 + i * 8192, R, C);
        voffA[i] = (unsigned)(R * KCH + C) * 2u;
        const int r = wid * 4 + (lane >> 4) + i * 32, c = (lane & 15) ^ (((r & 3) << 2) | ((r >> 3) & 1));
        voffB[i] = (unsigned)r * (unsigned)(NCOL * 2) + (unsigned)c * 16u;
    }
    const unsigned ldsw = (unsigned)wid * 1024u;
    const int aoff = lds_byte(wr * 64 + fr, fq * 8);
    const int q4 = (lane & 15) >> 2, p4 = lane & 3;
    const int boff0 = 256 * (8 * fq + q4) + 16 * ((4 * wc + p4) ^ (q4 << 2) ^ (fq & 1)) + 8 * (p4 & 1);
    const unsigned bad0 = (unsigned)(size_t)lds + 65536u + (unsigned)boff0, bad1 = bad0 ^ 8u;

    const int flip = h ^ 1;
    const long aoffs0 = (long)(m * 256 + flip * 128) * (KCH * 2) + h * 512;
    const long hstepA = (flip ? -1L : 1L) * 128 * (KCH * 2);
    const char* xloc = x + (size_t)rank * BLK_BYTES + aoffs0;
    const char* wbase = w + (size_t)(h * 256) * (NCOL * 2) + (size_t)ntile * 512;
    constexpr long hstepB = 256;

    const int coff = COFF(h * 2 + m);
#define TILE_PTRS(u_, A_, B_) do { const int _u = (u_); const int _j = (_u * 37) >> 8; \
        const int _d0 = _u - _j * 7 + coff; const int _d = _d0 >= 7 ? _d0 - 7 : _d0; const int _p = (rank + 1 + _d) & 7; \
        A_ = xrem + (size_t)_j * BLK_BYTES + _p * 128; \
        B_ = wbase + (size_t)(_p * KCH + _j * 64) * (NCOL * 2); } while (0)

#define SA(bb, hh) (((bb) * 2 + (hh)) * HTB)
#define SB(bb, hh) ((4 + (bb) * 2 + (hh)) * HTB)
#define WAUX 0
#define GLDS(gp, loff, aux) __builtin_amdgcn_global_load_lds((const unsigned*)(gp), (LAS unsigned*)(lds + (loff)), 16, 0, aux)
#define STAGE(bufoff, gbase, voff, aux) do { GLDS((gbase) + (voff)[0], (bufoff) + ldsw, aux); GLDS((gbase) + (voff)[1], (bufoff) + ldsw + 8192, aux); } while (0)
#define STA(bb, hh, ap) STAGE(SA(bb, hh), (ap) + (hh) * hstepA, voffA, 0)
#define STB(bb, hh, bp) STAGE(SB(bb, hh), (bp) + (hh) * hstepB, voffB, WAUX)
#define LDA(dst, bb, hh) do { _Pragma("unroll") for (int mm = 0; mm < 4; ++mm) _Pragma("unroll") for (int k = 0; k < 2; ++k) \
        dst[mm][k] = *(const LAS bf16x8*)(lds + SA(bb, hh) + aoff + mm * 2048 + k * 1024); } while (0)
#define TRD(dst, base, off) asm volatile("ds_read_b64_tr_b16 %0, %1 offset:%2" : "=v"(dst) : "v"(base), "n"(off) : "memory")
#define LDB(dst, bb, hh) do { _Pragma("unroll") for (int n = 0; n < 2; ++n) _Pragma("unroll") for (int k = 0; k < 2; ++k) { \
        s16x4 _lo, _hi; TRD(_lo, (n ? bad1 : bad0), SB(bb, hh) - 65536 + k * 8192); TRD(_hi, (n ? bad1 : bad0), SB(bb, hh) - 65536 + k * 8192 + 1024); \
        dst[n][k] = __builtin_shufflevector(_lo, _hi, 0, 1, 2, 3, 4, 5, 6, 7); } } while (0)
#define WAIT_LB(Bt_) asm volatile("s_waitcnt lgkmcnt(0)" : "+v"(Bt_[0][0]), "+v"(Bt_[0][1]), "+v"(Bt_[1][0]), "+v"(Bt_[1][1]) :: "memory")
#define MMA(ai, bj, At_, Bt_) do { __builtin_amdgcn_s_setprio(1); _Pragma("unroll") for (int mm = 0; mm < 4; ++mm) _Pragma("unroll") for (int n = 0; n < 2; ++n) _Pragma("unroll") for (int k = 0; k < 2; ++k) \
        acc[ai][bj][mm][n] = __builtin_amdgcn_mfma_f32_16x16x32_bf16(Bt_[n][k], At_[mm][k], acc[ai][bj][mm][n], 0, 0, 0); __builtin_amdgcn_s_setprio(0); } while (0)

#define EARLYB 0
#define MMAB(ai, bj, At_, Bt_) do { __builtin_amdgcn_s_setprio(1); _Pragma("unroll") for (int mm = 0; mm < 4; ++mm) { \
        if (mm == 4 - EARLYB) { SCHED; BAR; SCHED; } \
        _Pragma("unroll") for (int n = 0; n < 2; ++n) _Pragma("unroll") for (int k = 0; k < 2; ++k) \
        acc[ai][bj][mm][n] = __builtin_amdgcn_mfma_f32_16x16x32_bf16(Bt_[n][k], At_[mm][k], acc[ai][bj][mm][n], 0, 0, 0); } \
        __builtin_amdgcn_s_setprio(0); if (EARLYB == 0) BAR; SCHED; } while (0)

    const char *a0 = xloc, *b0 = wbase + (size_t)(rank * KCH) * (NCOL * 2), *a1 = a0 + 128, *b1 = b0 + (size_t)64 * (NCOL * 2);
    const bool pusher = b < 224;
    const int k1 = (b >> 2) & 7;
    const unsigned dst = (unsigned)(rank + 1 + (b >> 5)) & 7u;
    int pcl = 0, uc = 0;
    {
        int u[4];
#pragma unroll
        for (int c = 0; c < 4; ++c) { const int v = 6 - (b >> 5) - COFF(c); u[c] = v < 0 ? v + 7 : v; }
#pragma unroll
        for (int c = 0; c < 4; ++c) {
            const int o = (u[0] * 4 + 0 < u[c] * 4 + c) + (u[1] * 4 + 1 < u[c] * 4 + c) + (u[2] * 4 + 2 < u[c] * 4 + c) + (u[3] * 4 + 3 < u[c] * 4 + c);
            if (o == (k1 & 3)) { pcl = c; uc = u[c]; }
        }
    }
    const int pj = k1 >> 2;
    const unsigned loff = (unsigned)((pcl & 1) * 256 + (b & 3) * 64 + wid * 8 + (lane >> 3)) * (KCH * 2) + (lane & 7) * 16;
    const char* psrc = x + (size_t)dst * BLK_BYTES + (pcl >> 1) * 512 + pj * 128;
    u32x4 pv;
    if (pusher) asm volatile("global_load_dwordx4 %0, %1, %2 nt" : "=v"(pv) : "v"(loff), "s"(psrc) : "memory");
    STB(0, 0, b0); STA(0, 0, a0); STB(0, 1, b0); STA(0, 1, a0); STB(1, 0, b1); STA(1, 0, a1); STB(1, 1, b1);
    char* pws = (char*)ws_all[dst];
    WAIT_V(14);
    asm volatile("" : "+v"(cntv));
    const unsigned call = (unsigned)__builtin_amdgcn_readfirstlane(cntv) + 1u;
    const unsigned par = call & 1u;
    const char* xrem = ws + WS_RECV + (size_t)(par * 8 + h * 4) * BLK_BYTES + (size_t)(m * 256 + flip * 128) * (KCH * 2);
    char* pdst = pws + WS_RECV + (size_t)(par * 8 + (pcl >> 1) * 4 + pj) * BLK_BYTES + rank * 128;
    char* pflag = pws + WS_FLAGA + (size_t)(par * 16384 + pcl * 4096 + (7 * pj + uc) * 128 + (b & 3) * 4);
    bool fl = false;
    int ps = 0, ev = pusher ? (k1 ? k1 : 1) : 1000;
#define PUSH_SLOT() do { if (--ev == 0) { \
        if (ps & 1) { WAIT_V(0); fl = wid == 4; } \
        else { asm volatile("global_store_dwordx4 %0, %1, %2 sc0 sc1\n\ts_nop 2" :: "v"(loff), "v"(pv), "s"(pdst) : "memory"); pdst += 2 * BLK_BYTES; \
               if (ps == 0) asm volatile("global_load_dwordx4 %0, %1, %2 offset:256 nt" : "+v"(pv) : "v"(loff), "s"(psrc) : "memory"); } \
        ++ps; ev = ps < 4 ? (ps == 1 ? GAP1 : 4) : 1000; } } while (0)
#define FLAG_SLOT() do { if (fl) { fl = false; if (lane == 0) asm volatile("global_store_dword %0, %1, %2 sc0 sc1" :: "v"(0u), "v"(call), "s"(pflag) : "memory"); pflag += 14 * 128; } } while (0)
    PUSH_SLOT();

    f32x4 acc[2][2][4][2];
#pragma unroll
    for (int a = 0; a < 2; ++a)
#pragma unroll
        for (int c = 0; c < 2; ++c)
#pragma unroll
            for (int mm = 0; mm < 4; ++mm)
#pragma unroll
                for (int n = 0; n < 2; ++n) acc[a][c][mm][n] = (f32x4){0.f, 0.f, 0.f, 0.f};
    bf16x8 At[4][2], B0[2][2], B1[2][2];
    bool good = true;
    const char* fbase = ws + WS_FLAGA + par * 16384 + (h * 2 + m) * 4096;
    const unsigned fo = (unsigned)(lane & 31) * 128u, cc = call;
    constexpr unsigned FULL = 0x0FFFFFFFu;
    unsigned rdy = wid == 0 ? 0u : FULL, used = 0u, late = 0u;
    u32x4 pa = {cc ^ 1u, 0u, 0u, 0u}, pb = pa;
    unsigned dv = 0u;
#define POLL_ISSUE(pv_) do { if (rdy != FULL) asm volatile("global_load_dwordx4 %0, %1, %2 sc0 sc1" : "+v"(pv_) : "v"(fo), "s"(fbase) : "memory"); } while (0)
#define POLL_MASK(pv_) do { asm volatile("" : "+v"(pv_)); \
        const unsigned long long _m = __builtin_amdgcn_ballot_w64(((pv_[0] ^ cc) | (pv_[1] ^ cc) | (pv_[2] ^ cc) | (pv_[3] ^ cc)) == 0u); \
        rdy |= (unsigned)_m & FULL; } while (0)
#define FOLD(z) (((z) | ((z) >> 7) | ((z) >> 14) | ((z) >> 21)) & 0x7Fu)
#define DECIDE(pv_, slot) do { if (wid == 0 && used != FULL) { \
        const unsigned _cand = FULL & ~used; \
        if (rdy != FULL) { POLL_MASK(pv_); \
            unsigned _sel = _cand & (rdy | ~late); \
            if ((_sel & -_sel & rdy) == 0u) { const unsigned long long _t0 = __builtin_amdgcn_s_memrealtime(); unsigned long long _t1 = 0ull; \
                for (;;) { POLL_ISSUE(pv_); WAIT_V(0); POLL_MASK(pv_); \
                    _sel = _cand & (rdy | ~late); if ((_sel & -_sel & rdy) != 0u) break; \
                    const unsigned long long _now = __builtin_amdgcn_s_memrealtime(); const unsigned _av = rdy & _cand; \
                    if (_av != 0u) { if (_t1 == 0ull) _t1 = _now; \
                        if (_now - _t1 >= PAT) { late |= (FOLD(_cand & ((_av & -_av) - 1u)) | (LATEALL ? FOLD(rdy) ^ 0x7Fu : 0u)) * 0x204081u; break; } } \
                    if ((unsigned)__builtin_amdgcn_readfirstlane(__hip_atomic_load(tmo, __ATOMIC_RELAXED, __HIP_MEMORY_SCOPE_AGENT)) == call || \
                        _now - _t0 > DEADLINE_TICKS) { \
                        __hip_atomic_store(tmo, call, __ATOMIC_RELAXED, __HIP_MEMORY_SCOPE_AGENT); good = false; rdy = FULL; break; } \
                    __builtin_amdgcn_s_sleep(1); } } } \
        const int _u2 = __builtin_ctz(rdy & _cand); used |= 1u << _u2; \
        asm volatile("ds_write_b32 %0, %1 offset:%2\n\ts_waitcnt lgkmcnt(0)" :: "v"(ldsd), "v"(_u2), "n"((slot) * 4) : "memory"); } } while (0)
#define DEC_READ(slot) asm volatile("ds_read_b32 %0, %1 offset:%2" : "=v"(dv) : "v"(ldsd), "n"((slot) * 4) : "memory")
#define DEC_PTRS(A_, B_) do { asm volatile("" : "+v"(dv)); TILE_PTRS(__builtin_amdgcn_readfirstlane(dv), A_, B_); } while (0)
    const unsigned ldsd = (unsigned)(size_t)lds + (unsigned)LDS_DEC;

    const char *a2 = a0 + 256, *b2 = b0 + (size_t)128 * (NCOL * 2), *a3 = a1, *b3 = b0 + (size_t)192 * (NCOL * 2);
    const char* sbase = ws + WS_SYNC;
    unsigned sv = 0u, sval = call * 16u;
#define PAIR_POST() do { ++sval; if (wid == 0 && lane == 0) asm volatile("global_store_dword %0, %1, %2 sc1" :: "v"(b * 128), "v"(sval), "s"(sbase) : "memory"); } while (0)
#define PAIR_ISSUE() do { if (wid == 0) asm volatile("global_load_dword %0, %1, %2 sc1" : "+v"(sv) : "v"((b ^ 8) * 128), "s"(sbase) : "memory"); } while (0)
#define PAIR_CHECK() do { if (wid == 0 && t >= 2) { asm volatile("" : "+v"(sv)); \
        if ((int)((unsigned)__builtin_amdgcn_readfirstlane(sv) - (sval - 1u)) < 0) good = wait_flags((const unsigned*)(sbase + (b ^ 8) * 128), sval, call, tmo) && good; } } while (0)
    WAIT_V(10); BAR;
    if (wr == 1) BAR;
    STAMP(1);

    LDB(B0, 0, 0);
#define ITER(PUSH) do { \
        if (PUSH) FLAG_SLOT(); PAIR_POST(); DEC_READ(0); LDA(At, 0, 0); STA(1, 1, a3); \
        WAIT_V(10); WAIT_L(8); BAR; WAIT_LB(B0); MMAB(0, 0, At, B0); \
        if (PUSH) PUSH_SLOT(); LDB(B1, 0, 1); if (!PUSH || t >= 2) DEC_PTRS(a2, b2); STB(0, 0, b2); \
        WAIT_V(10); BAR; WAIT_LB(B1); MMAB(0, 1, At, B1); \
        if (PUSH) FLAG_SLOT(); POLL_ISSUE(pa); LDA(At, 0, 1); STA(0, 0, a2); \
        WAIT_V(10); BAR; WAIT_L(0); MMAB(1, 0, At, B0); \
        if (PUSH) PUSH_SLOT(); if (!PUSH || t >= 2) DECIDE(pb, 1); LDB(B0, 1, 0); STB(0, 1, b2); \
        WAIT_V(10); BAR; MMAB(1, 1, At, B1); \
        if (PUSH) FLAG_SLOT(); PAIR_CHECK(); DEC_READ(1); LDA(At, 1, 0); STA(0, 1, a2); \
        WAIT_V(10); WAIT_L(8); BAR; WAIT_LB(B0); MMAB(0, 0, At, B0); \
        if (PUSH) PUSH_SLOT(); LDB(B1, 1, 1); if (!PUSH || t >= 2) DEC_PTRS(a3, b3); else a3 = a0 + 384; STB(1, 0, b3); \
        WAIT_V(10); BAR; WAIT_LB(B1); MMAB(0, 1, At, B1); \
        if (PUSH) FLAG_SLOT(); POLL_ISSUE(pb); LDA(At, 1, 1); PAIR_ISSUE(); STA(1, 0, a3); \
        WAIT_V(10); BAR; WAIT_L(0); MMAB(1, 0, At, B0); \
        if (PUSH) PUSH_SLOT(); DECIDE(pa, 0); LDB(B0, 0, 0); STB(1, 1, b3); \
        WAIT_V(10); BAR; MMAB(1, 1, At, B1); \
        } while (0)

#pragma unroll 1
    for (int t = 0; t < TPUSH; t += 2) {
        if (t == 2) STAMP(2);
        if (t == 4) STAMP(3);
        if (t == 6) { STAMP(4); STAMPC(14); }
        if (t == 8) STAMP(5);
        ITER(1);
    }
#pragma unroll 1
    for (int t = TPUSH; t < NT - 2; t += 2) {
        if (t == 16) { STAMP(6); STAMPC(15); }
        if (t == 20) { STAMP(7); STAMPC(12); }
        ITER(0);
    }
    STAMP(8); STAMPC(13);

    int tz = threadIdx.x; asm volatile("" : "+v"(tz));
    const int lane2 = tz & 63, fr2 = lane2 & 15, fq2 = lane2 >> 4;
    {
        char* xs = ws + WS_XCH + ((size_t)b * 8 + wid) * 8192 + lane2 * 16;
#define XSEND(c) do { _Pragma("unroll") for (int mm = 0; mm < 4; ++mm) { \
            const f32x4 u0 = acc[0][c][mm][0], u1 = acc[0][c][mm][1]; \
            const u32x4 o = {cvt_pk_bf16(u0[0], u0[1]), cvt_pk_bf16(u0[2], u0[3]), cvt_pk_bf16(u1[0], u1[1]), cvt_pk_bf16(u1[2], u1[3])}; \
            asm volatile("global_store_dwordx4 %0, %1, off offset:%2 sc1\n\ts_nop 2" :: "v"(xs + (c) * 4096), "v"(o), "n"(mm * 1024) : "memory"); } } while (0)
        LDA(At, 0, 0); STA(1, 1, a3);
        WAIT_V(10); BAR; WAIT_LB(B0); MMAB(0, 0, At, B0);
        LDB(B1, 0, 1);
        WAIT_V(8); BAR; WAIT_LB(B1); MMAB(0, 1, At, B1);
        LDA(At, 0, 1);
        WAIT_V(6); BAR; WAIT_L(0); MMAB(1, 0, At, B0);
        LDB(B0, 1, 0);
        WAIT_V(4); BAR; MMAB(1, 1, At, B1);
        LDA(At, 1, 0);
        WAIT_V(2); BAR; WAIT_LB(B0); MMAB(0, 0, At, B0);
        LDB(B1, 1, 1); WAIT_V(0); XSEND(0); SCHED;
        BAR; WAIT_LB(B1); MMAB(0, 1, At, B1);
        LDA(At, 1, 1); XSEND(1); SCHED;
        BAR; WAIT_L(0); MMA(1, 0, At, B0); if (wr == 0) BAR;
        SCHED; MMA(1, 1, At, B1); SCHED;
        WAIT_V(0);
        if (lane2 == 0) __hip_atomic_store((unsigned*)(ws + WS_FLAGX + ((size_t)b * 8 + wid) * 128), call, __ATOMIC_RELAXED, __HIP_MEMORY_SCOPE_AGENT);
    }
    if (tz == 0) __hip_atomic_store(cnt, call, __ATOMIC_RELAXED, __HIP_MEMORY_SCOPE_AGENT);

    STAMP(9);
    {
        const int pb = b ^ 16;
        const unsigned* xf = (const unsigned*)(ws + WS_FLAGX + ((size_t)pb * 8 + wid) * 128);
        const unsigned long long t0 = __builtin_amdgcn_s_memrealtime();
        while ((int)((unsigned)__builtin_amdgcn_readfirstlane(__hip_atomic_load(xf, __ATOMIC_RELAXED, __HIP_MEMORY_SCOPE_AGENT)) - call) < 0) {
            if ((unsigned)__builtin_amdgcn_readfirstlane(__hip_atomic_load(tmo, __ATOMIC_RELAXED, __HIP_MEMORY_SCOPE_AGENT)) == call ||
                __builtin_amdgcn_s_memrealtime() - t0 > DEADLINE_TICKS) {
                __hip_atomic_store(tmo, call, __ATOMIC_RELAXED, __HIP_MEMORY_SCOPE_AGENT);
                break;
            }
            __builtin_amdgcn_s_sleep(1);
        }
        STAMP(10);
        const char* xr = ws + WS_XCH + ((size_t)pb * 8 + wid) * 8192 + lane2 * 16;
        u32x4 rv[2][4];
#pragma unroll
        for (int mm = 0; mm < 4; ++mm)
#pragma unroll
            for (int c = 0; c < 2; ++c)
                asm volatile("global_load_dwordx4 %0, %1, off sc1" : "=v"(rv[c][mm]) : "v"(xr + (c * 4 + mm) * 1024) : "memory");
        const bool bad = !good || (unsigned)__builtin_amdgcn_readfirstlane(__hip_atomic_load(tmo, __ATOMIC_RELAXED, __HIP_MEMORY_SCOPE_AGENT)) == call;
        WAIT_V(0);
#pragma unroll
        for (int c = 0; c < 2; ++c)
#pragma unroll
            for (int mm = 0; mm < 4; ++mm) asm volatile("" : "+v"(rv[c][mm]));
        char* op = out + (size_t)(m * 256 + h * 128 + wr * 64 + fr2) * (NCOL * 2) + (size_t)(ntile * 256 + wc * 32 + 8 * fq2) * 2;
        const bool odd = fq2 & 1;
#pragma unroll
        for (int mm = 0; mm < 4; ++mm)
#pragma unroll
            for (int c = 0; c < 2; ++c) {
                const u32x4 r = rv[c][mm];
                const f32x4 v0 = acc[1][c][mm][0] + (f32x4){__uint_as_float(r[0] << 16), __uint_as_float(r[0] & 0xFFFF0000u), __uint_as_float(r[1] << 16), __uint_as_float(r[1] & 0xFFFF0000u)};
                const f32x4 v1 = acc[1][c][mm][1] + (f32x4){__uint_as_float(r[2] << 16), __uint_as_float(r[2] & 0xFFFF0000u), __uint_as_float(r[3] << 16), __uint_as_float(r[3] & 0xFFFF0000u)};
                const unsigned l0 = cvt_pk_bf16(silu(v0[0]), silu(v0[1])), l1 = cvt_pk_bf16(silu(v0[2]), silu(v0[3]));
                const unsigned h0 = cvt_pk_bf16(silu(v1[0]), silu(v1[1])), h1 = cvt_pk_bf16(silu(v1[2]), silu(v1[3]));
                u32x4 o = odd ? (u32x4){h0, h1, l0, l1} : (u32x4){l0, l1, h0, h1};
                if (bad) o = (u32x4){0x7FC07FC0u, 0x7FC07FC0u, 0x7FC07FC0u, 0x7FC07FC0u};
                *(u32x4*)(op + (size_t)mm * 16 * (NCOL * 2) + c * 256) = o;
            }
        STAMP(11);
    }
}

extern "C" void kernel_launch(int rank, int nranks, ncclComm_t comm,
                              hipStream_t stream,
                              void* const* d_in, const int* in_sizes, int n_in,
                              void* d_out, int out_size,
                              const int* in_dtypes, int out_dtype,
                              void* const* d_out_all,
                              void* const* d_ws_all, size_t ws_bytes) {
    if (nranks != NR || (unsigned)rank >= (unsigned)NR || ws_bytes < WS_END) { fprintf(stderr, "unsupported configuration\n"); return; }
    static bool init = false;
    if (!init) {
        HIP_CHECK(hipFuncSetAttribute((const void*)a2a_gemm_silu, hipFuncAttributeMaxDynamicSharedMemorySize, LDS_BYTES));
        init = true;
    }
    hipLaunchKernelGGL(a2a_gemm_silu, dim3(NWG), dim3(512), LDS_BYTES, stream,
                       (const char*)d_in[0], (const char*)d_in[1], (char*)d_out, d_ws_all, rank);
    HIP_CHECK(hipGetLastError());
}
```
